```python
import numpy as np

kernel_name = "gpu_a2a_gemm_m4096_k4096_n4096_e4m3e5m2_dqrelu"


ABSMAX_RTOL = 0.05

COMPUTE_DTYPE = "float32"
EPILOGUE = "dequant_relu"
IN_QUANT = None
OUT_QUANT = None
IN_FP8 = "e4m3e5m2"
IN_DTYPES = {"x": "e4m3", "w_mat": "e5m2",
             "scale_x": "float32", "scale_w": "float32"}
WORLD_SIZE = 8

def _snap_fp8(x, mant_bits, max_abs, min_normal):
    a = np.abs(x.astype(np.float64))
    _m, _e = np.frexp(a)
    q = np.where(a >= min_normal, np.ldexp(1.0, _e - 1 - mant_bits),
                 min_normal * 2.0 ** -mant_bits)
    snapped = np.minimum(np.round(a / q) * q, max_abs)
    return (np.sign(x) * snapped).astype(np.float32)

def setup_inputs():

    x = _snap_fp8(np.random.randn(4096, 4096).astype(np.float32), 3, 448.0, 2.0 ** -6)
    w_mat = _snap_fp8(np.random.randn(4096, 4096).astype(np.float32), 2, 57344.0, 2.0 ** -14)
    scale_x = (0.02 * (1.0 + np.random.rand(1))).astype(np.float32)
    scale_w = (0.02 * (1.0 + np.random.rand(1))).astype(np.float32)
    return {"x": x, "w_mat": w_mat, "scale_x": scale_x, "scale_w": scale_w}

def reference(x, w_mat, scale_x, scale_w):
    acc = (x @ w_mat).astype(np.float32)
    y = (acc * (scale_x * scale_w)).astype(np.float32)
    return np.maximum(y, 0.0).astype(np.float32)
```

```cpp
#include <hip/hip_runtime.h>
#include <rccl/rccl.h>
#include <cstdio>
#include <cstdint>

typedef int v8i __attribute__((ext_vector_type(8)));
typedef int v4i __attribute__((ext_vector_type(4)));
typedef int v2i __attribute__((ext_vector_type(2)));
typedef float v4f __attribute__((ext_vector_type(4)));
typedef __attribute__((address_space(3))) unsigned char lds_u8;
typedef __attribute__((address_space(3))) unsigned lds_u32;
typedef __attribute__((address_space(3))) v4i lds_v4i;
typedef __attribute__((address_space(3))) v2i lds_v2i;

namespace {
constexpr int NR = 8;
constexpr int KC = 512;
constexpr int NTOT = 4096;
constexpr int BM = 64, BN = 128, BK = 128;
constexpr int NSTAGE = 32;
constexpr int NS = 6;
constexpr int SLOT_A = BM * BK, SLOT_B = BK * BN, SLOT = SLOT_A + SLOT_B;
constexpr int CTRL = NS * SLOT;
constexpr int C_READY = CTRL;
constexpr int C_CALL = CTRL + 8;
constexpr int C_BAD = CTRL + 12;
constexpr int C_FLAGS = CTRL + 256;
constexpr int LDS_BYTES = CTRL + 512;
constexpr int NWG = 256;
constexpr int NTHREADS = 384;

constexpr size_t WS_STAGE = 0;
constexpr size_t STAGE_SRC = 512 * 512;
constexpr size_t STAGE_PAR = NR * STAGE_SRC;
constexpr size_t WS_FLAGS = 2 * STAGE_PAR;
constexpr size_t FLAG_STRIDE = 128;
constexpr size_t WS_CNT = WS_FLAGS + NR * 32 * FLAG_STRIDE;
constexpr size_t WS_TMO = WS_CNT + NWG * 64;
constexpr size_t WS_TOTAL = WS_TMO + 128;
constexpr unsigned long long DEADLINE_TICKS = 50000000ull;

struct Params {
  const unsigned char* x;
  const unsigned char* w;
  const float* sx;
  const float* sw;
  float* out;
  unsigned char* ws[NR];
  int rank;
};

__device__ __forceinline__ unsigned vmcnt_now() {
  unsigned s = __builtin_amdgcn_s_getreg(7 | (0 << 6) | (31 << 11));
  return (s & 15u) | ((s >> 18) & 0x30u);
}

__device__ __forceinline__ void wait_vm(int n) {
  switch (n) {
    case 0: asm volatile("s_waitcnt vmcnt(0)" ::: "memory"); break;
    case 1: asm volatile("s_waitcnt vmcnt(6)" ::: "memory"); break;
    case 2: asm volatile("s_waitcnt vmcnt(12)" ::: "memory"); break;
    case 3: asm volatile("s_waitcnt vmcnt(18)" ::: "memory"); break;
    case 4: asm volatile("s_waitcnt vmcnt(24)" ::: "memory"); break;
    default: asm volatile("s_waitcnt vmcnt(30)" ::: "memory"); break;
  }
}

__global__ void __launch_bounds__(NTHREADS, 2) fused_a2a_gemm(Params p) {
  extern __shared__ __attribute__((aligned(16))) unsigned char smem_generic[];
  lds_u8* smem = (lds_u8*)smem_generic;
  const int tid = threadIdx.x;
  const int wave = __builtin_amdgcn_readfirstlane(tid >> 6);
  const int lane = tid & 63;
  const int b = blockIdx.x;
  const int rank = p.rank;
  const int xcd = b & 7, idx = b >> 3;
  const int ntile = xcd * 4 + (idx & 3);
  const int mb = idx >> 2;
  unsigned char* myws = p.ws[rank];
  unsigned* cnt_ptr = (unsigned*)(myws + WS_CNT + (size_t)b * 64);

  if (wave < 4) {
    const int w = wave;
    const int g = lane >> 4, i16 = lane & 15;
    unsigned offA[2], offB[4];
#pragma unroll
    for (int i = 0; i < 2; ++i) {
      int row = (2 * w + i) * 8 + (lane >> 3);
      offA[i] = row * KC + (((lane & 7) ^ ((row >> 1) & 7)) * 16);
    }
#pragma unroll
    for (int i = 0; i < 4; ++i) {
      int kr = (4 * w + i) * 8 + (lane >> 3);
      int f = ((kr >> 1) & 3) | ((kr >> 2) & 4);
      offB[i] = kr * NTOT + (((lane & 7) ^ f) * 16) + ntile * BN;
    }
    unsigned rdA[2];
    {
      int sw = (i16 >> 1) & 7;
      rdA[0] = i16 * 128 + ((g ^ sw) * 16);
      rdA[1] = i16 * 128 + (((4 + g) ^ sw) * 16);
    }
    unsigned rdB[2];
    {
      int k = 16 * g + (i16 >> 1);
      int f = ((k >> 1) & 3) | ((k >> 2) & 4);
#pragma unroll
      for (int nt = 0; nt < 2; ++nt)
        rdB[nt] = SLOT_A + k * 128 + (((2 * w + nt) ^ f) * 16) + (i16 & 1) * 8;
    }
    v4f acc[4][2];
#pragma unroll
    for (int mt = 0; mt < 4; ++mt)
#pragma unroll
      for (int nt = 0; nt < 2; ++nt) acc[mt][nt] = v4f{0.f, 0.f, 0.f, 0.f};

    const unsigned char* own_a = p.x + (size_t)(rank * 512 + mb * BM) * KC;
    const unsigned char* stg_a = nullptr;

    auto issue = [&](int j) {
      const unsigned char* ab;
      int kg;
      if (j < 4) {
        ab = own_a + j * BK;
        kg = rank * KC + j * BK;
      } else {
        int s = (j - 4) / 7, t = (j - 4) - 7 * s;
        int src = (rank + 1 + t) & 7;
        ab = stg_a + (size_t)src * STAGE_SRC + s * BK;
        kg = src * KC + s * BK;
      }
      const unsigned char* bb = p.w + (size_t)kg * NTOT;
      int slot = j % NS;
      lds_u8* la = smem + slot * SLOT + (2 * w) * 1024;
      lds_u8* lb = smem + slot * SLOT + SLOT_A + (4 * w) * 1024;
#pragma unroll
      for (int i = 0; i < 2; ++i)
        __builtin_amdgcn_global_load_lds((const unsigned*)(ab + offA[i]), (lds_u32*)(la + i * 1024), 16, 0, 17);
#pragma unroll
      for (int i = 0; i < 4; ++i)
        __builtin_amdgcn_global_load_lds((const unsigned*)(bb + offB[i]), (lds_u32*)(lb + i * 1024), 16, 0, 0);
    };

    int c = 0, iss = 0, it = 0;
#pragma unroll
    for (int j = 0; j < 4; ++j) issue(j);
    iss = 4;
    for (;;) {
      const bool comp = c < iss;
      if (comp) wait_vm(iss - c - 1);
      __builtin_amdgcn_s_barrier();
      int rdy = __builtin_amdgcn_readfirstlane(*(volatile lds_u32*)(smem + C_READY + (it & 1) * 4));
      if (it == 0) {
        unsigned call = __builtin_amdgcn_readfirstlane(*(volatile lds_u32*)(smem + C_CALL));
        stg_a = myws + WS_STAGE + (size_t)(call & 1) * STAGE_PAR + (size_t)mb * BM * KC;
      }
      int lim = min(rdy, c + NS);
      while (iss < lim) { issue(iss); ++iss; }
      if (comp) {
        const unsigned sb = (unsigned)((c % NS) * SLOT);
        const unsigned aB0 = sb + rdB[0], aB1 = sb + rdB[1], aA0 = sb + rdA[0], aA1 = sb + rdA[1];
        v2i b00, b01, b02, b03, b10, b11, b12, b13;
        v4i a00, a01, a10, a11, a20, a21, a30, a31;
#define TR8(dst, ad, o) asm volatile("ds_read_b64_tr_b8 %0, %1 offset:" #o : "=v"(dst) : "v"(ad))
#define RD128(dst, ad, o) asm volatile("ds_read_b128 %0, %1 offset:" #o : "=v"(dst) : "v"(ad))
        TR8(b00, aB0, 0); TR8(b01, aB0, 1024); TR8(b02, aB0, 8192); TR8(b03, aB0, 9216);
        RD128(a00, aA0, 0); RD128(a01, aA1, 0);
        TR8(b10, aB1, 0); TR8(b11, aB1, 1024); TR8(b12, aB1, 8192); TR8(b13, aB1, 9216);
        RD128(a10, aA0, 2048); RD128(a11, aA1, 2048);
        RD128(a20, aA0, 4096); RD128(a21, aA1, 4096);
        RD128(a30, aA0, 6144); RD128(a31, aA1, 6144);
#define MK8(x, y) v8i{x[0], x[1], x[2], x[3], y[0], y[1], y[2], y[3]}
#define MKB(p0, p1, p2, p3) v8i{p0[0], p0[1], p1[0], p1[1], p2[0], p2[1], p3[0], p3[1]}
#define MMA(d, wv, xv) d = __builtin_amdgcn_mfma_scale_f32_16x16x128_f8f6f4(wv, xv, d, 1, 0, 0, 0x7f7f7f7f, 0, 0x7f7f7f7f)
        asm volatile("s_waitcnt lgkmcnt(10)" : "+v"(b00), "+v"(b01), "+v"(b02), "+v"(b03), "+v"(a00), "+v"(a01));
        const v8i w0 = MKB(b00, b01, b02, b03);
        const v8i x0 = MK8(a00, a01);
        MMA(acc[0][0], w0, x0);
        asm volatile("s_waitcnt lgkmcnt(6)" : "+v"(b10), "+v"(b11), "+v"(b12), "+v"(b13));
        const v8i w1 = MKB(b10, b11, b12, b13);
        MMA(acc[0][1], w1, x0);
        asm volatile("s_waitcnt lgkmcnt(4)" : "+v"(a10), "+v"(a11));
        const v8i x1 = MK8(a10, a11);
        MMA(acc[1][0], w0, x1);
        MMA(acc[1][1], w1, x1);
        asm volatile("s_waitcnt lgkmcnt(2)" : "+v"(a20), "+v"(a21));
        const v8i x2 = MK8(a20, a21);
        MMA(acc[2][0], w0, x2);
        MMA(acc[2][1], w1, x2);
        asm volatile("s_waitcnt lgkmcnt(0)" : "+v"(a30), "+v"(a31));
        const v8i x3 = MK8(a30, a31);
        MMA(acc[3][0], w0, x3);
        MMA(acc[3][1], w1, x3);
        ++c;
      }
      ++it;
      if (c == NSTAGE) break;
    }
    const unsigned bad = __builtin_amdgcn_readfirstlane(*(volatile lds_u32*)(smem + C_BAD));
    float sc = p.sx[0] * p.sw[0];
    float* ob = p.out + (size_t)(mb * BM + i16) * NTOT + ntile * BN + w * 32 + g * 4;
#pragma unroll
    for (int mt = 0; mt < 4; ++mt)
#pragma unroll
      for (int nt = 0; nt < 2; ++nt) {
        v4f v = acc[mt][nt];
        v4f y;
#pragma unroll
        for (int r = 0; r < 4; ++r) y[r] = bad ? __builtin_nanf("") : fmaxf(v[r] * sc, 0.f);
        *(v4f*)(ob + (size_t)mt * 16 * NTOT + nt * 16) = y;
      }
  } else if (wave == 4) {
    const int dst = b & 7, rg = b >> 3;
    const bool active = dst != rank;
    unsigned call = __hip_atomic_load(cnt_ptr, __ATOMIC_RELAXED, __HIP_MEMORY_SCOPE_AGENT) + 1u;
    call = __builtin_amdgcn_readfirstlane(call);
    const unsigned base = (call - 1u) * 4u;
    unsigned* flag = (unsigned*)(p.ws[dst] + WS_FLAGS + (size_t)(rank * 32 + rg) * FLAG_STRIDE);
    if (active) {
      const unsigned char* src = p.x + (size_t)(dst * 512 + rg * 16) * KC;
      unsigned char* dp = p.ws[dst] + WS_STAGE + (size_t)(call & 1) * STAGE_PAR + (size_t)rank * STAGE_SRC + (size_t)rg * 16 * KC;
      v4i d[8];
      unsigned off[8];
#pragma unroll
      for (int j = 0; j < 8; ++j) {
        off[j] = ((j & 1) * 8 + (lane >> 3)) * KC + (j >> 1) * BK + (lane & 7) * 16;
        d[j] = *(const v4i*)(src + off[j]);
      }
      asm volatile("" : "+v"(d[0]), "+v"(d[1]), "+v"(d[2]), "+v"(d[3]), "+v"(d[4]), "+v"(d[5]), "+v"(d[6]), "+v"(d[7]));
      asm volatile("s_waitcnt vmcnt(0)" ::: "memory");
#pragma unroll
      for (int j = 0; j < 8; ++j)
        asm volatile("global_store_dwordx4 %0, %1, off sc0 sc1\n\ts_nop 1" ::"v"(dp + off[j]), "v"(d[j]) : "memory");
    }
    int sent = active ? 0 : 4;
    int nissued = 8;
    int c = 0, iss = 4, it = 0;
    for (;;) {
      const bool comp = c < iss;
      if (sent < 4) {
        int done = nissued - (int)vmcnt_now();
        int lvl = min(done >> 1, 4);
        if (lvl > sent) {
          if (lane == 0) __hip_atomic_store(flag, base + lvl, __ATOMIC_RELAXED, __HIP_MEMORY_SCOPE_SYSTEM);
          sent = lvl;
          ++nissued;
        }
      }
      __builtin_amdgcn_s_barrier();
      int rdy = __builtin_amdgcn_readfirstlane(*(volatile lds_u32*)(smem + C_READY + (it & 1) * 4));
      int lim = min(rdy, c + NS);
      if (iss < lim) iss = lim;
      if (comp) ++c;
      ++it;
      if (c == NSTAGE) break;
    }
    if (sent < 4) {
      asm volatile("s_waitcnt vmcnt(0)" ::: "memory");
      if (lane == 0) __hip_atomic_store(flag, base + 4u, __ATOMIC_RELAXED, __HIP_MEMORY_SCOPE_SYSTEM);
    }
  } else {
    unsigned call = __hip_atomic_load(cnt_ptr, __ATOMIC_RELAXED, __HIP_MEMORY_SCOPE_AGENT) + 1u;
    call = __builtin_amdgcn_readfirstlane(call);
    const unsigned base = (call - 1u) * 4u;
    if (lane == 0) {
      *(volatile lds_u32*)(smem + C_CALL) = call;
      *(volatile lds_u32*)(smem + C_BAD) = 0u;
    }
    const unsigned char* fp = myws + WS_FLAGS + (size_t)(((lane >> 2) & 7) * 32 + mb * 4 + (lane & 3)) * FLAG_STRIDE;
    const int st_s = lane / 7, st_src = (rank + 1 + (lane - 7 * st_s)) & 7;
    const unsigned long long t0 = __builtin_amdgcn_s_memrealtime();
    int rdy = 4;
    bool pending = false;
    int c = 0, iss = 4, it = 0;
    for (;;) {
      const bool comp = c < iss;
      if (rdy < NSTAGE) {
        if (pending && vmcnt_now() == 0) {
          asm volatile("s_waitcnt vmcnt(0)" ::: "memory");
          int v = (int)(*(volatile lds_u32*)(smem + C_FLAGS + lane * 4) - base);
          v = min(v, __shfl_xor(v, 1));
          v = min(v, __shfl_xor(v, 2));
          int lv = __shfl(v, st_src * 4);
          unsigned long long ok = __ballot(lane < 28 && lv > st_s);
          rdy = 4 + __builtin_ctzll(~ok);
          pending = false;
          if (rdy < NSTAGE && __builtin_amdgcn_s_memrealtime() - t0 > DEADLINE_TICKS) {
            rdy = NSTAGE;
            if (lane == 0) {
              *(volatile lds_u32*)(smem + C_BAD) = 1u;
              __hip_atomic_store((unsigned*)(myws + WS_TMO), call, __ATOMIC_RELAXED, __HIP_MEMORY_SCOPE_AGENT);
            }
          }
        }
        if (!pending && rdy < NSTAGE) {
          __builtin_amdgcn_global_load_lds((const unsigned*)fp, (lds_u32*)(smem + C_FLAGS), 4, 0, 17);
          pending = true;
        }
      }
      if (lane == 0) *(volatile lds_u32*)(smem + C_READY + (it & 1) * 4) = (unsigned)rdy;
      asm volatile("s_waitcnt lgkmcnt(0)" ::: "memory");
      __builtin_amdgcn_s_barrier();
      int lim = min(rdy, c + NS);
      if (iss < lim) iss = lim;
      if (comp) ++c;
      ++it;
      if (c == NSTAGE) break;
    }
    if (lane == 0) __hip_atomic_store(cnt_ptr, call, __ATOMIC_RELAXED, __HIP_MEMORY_SCOPE_AGENT);
  }
}
}

extern "C" void kernel_launch(int rank, int nranks, ncclComm_t comm, hipStream_t stream,
                              void* const* d_in, const int* in_sizes, int n_in,
                              void* d_out, int out_size, const int* in_dtypes, int out_dtype,
                              void* const* d_out_all, void* const* d_ws_all, size_t ws_bytes) {
  if (nranks != NR || ws_bytes < WS_TOTAL) {
    fprintf(stderr, "unsupported configuration\n");
    return;
  }
  static bool attr_set = false;
  if (!attr_set) {
    (void)hipFuncSetAttribute((const void*)fused_a2a_gemm, hipFuncAttributeMaxDynamicSharedMemorySize, LDS_BYTES);
    attr_set = true;
  }
  Params p;
  p.x = (const unsigned char*)d_in[0];
  p.w = (const unsigned char*)d_in[1];
  p.sx = (const float*)d_in[2];
  p.sw = (const float*)d_in[3];
  p.out = (float*)d_out;
  for (int i = 0; i < NR; ++i) p.ws[i] = (unsigned char*)d_ws_all[i];
  p.rank = rank;
  hipLaunchKernelGGL(fused_a2a_gemm, dim3(NWG), dim3(NTHREADS), LDS_BYTES, stream, p);
}
```

```cpp
#include <hip/hip_runtime.h>
#include <rccl/rccl.h>
#include <cstdio>
#include <cstdint>

typedef int v8i __attribute__((ext_vector_type(8)));
typedef int v4i __attribute__((ext_vector_type(4)));
typedef int v2i __attribute__((ext_vector_type(2)));
typedef float v4f __attribute__((ext_vector_type(4)));
typedef __attribute__((address_space(3))) unsigned char lds_u8;
typedef __attribute__((address_space(3))) unsigned lds_u32;
typedef __attribute__((address_space(3))) v4i lds_v4i;
typedef __attribute__((address_space(3))) v2i lds_v2i;

#define AUX_A 17
namespace {
constexpr int NR = 8;
constexpr int KC = 512;
constexpr int NTOT = 4096;
constexpr int BM = 64, BN = 128, BK = 128;
constexpr int NSTAGE = 32;
constexpr int NS = 6;
constexpr int SLOT_A = BM * BK, SLOT_B = BK * BN, SLOT = SLOT_A + SLOT_B;
constexpr int CTRL = NS * SLOT;
constexpr int C_READY = CTRL;
constexpr int C_CALL = CTRL + 8;
constexpr int C_BAD = CTRL + 12;
constexpr int C_FLAGS = CTRL + 256;
constexpr int C_JUNK = CTRL + 512;
constexpr int LDS_BYTES = CTRL + 512 + 1024;
constexpr int NWG = 256;
constexpr int NTHREADS = 640;

constexpr size_t WS_STAGE = 0;
constexpr size_t STAGE_SRC = 512 * 512;
constexpr size_t STAGE_PAR = NR * STAGE_SRC;
constexpr size_t WS_FLAGS = 2 * STAGE_PAR;
constexpr size_t FLAG_STRIDE = 128;
constexpr size_t WS_CNT = WS_FLAGS + NR * 32 * FLAG_STRIDE;
constexpr size_t WS_TMO = WS_CNT + NWG * 64;
constexpr size_t WS_DBG = WS_TMO + 128;
constexpr size_t WS_TOTAL = WS_DBG + 256 * 64 * 8;
#define BARRIER() __builtin_amdgcn_s_barrier()
#define STAMP(i) do {} while (0)
constexpr unsigned long long DEADLINE_TICKS = 50000000ull;

struct Params {
  const unsigned char* x;
  const unsigned char* w;
  const float* sx;
  const float* sw;
  float* out;
  unsigned char* ws[NR];
  int rank;
};

__device__ __forceinline__ unsigned vmcnt_now() {
  unsigned s = __builtin_amdgcn_s_getreg(7 | (0 << 6) | (31 << 11));
  return (s & 15u) | ((s >> 18) & 0x30u);
}

__device__ __forceinline__ void wait_vm(int n) {
  switch (n) {
    case 0: asm volatile("s_waitcnt vmcnt(0)" ::: "memory"); break;
    case 1: asm volatile("s_waitcnt vmcnt(6)" ::: "memory"); break;
    case 2: asm volatile("s_waitcnt vmcnt(12)" ::: "memory"); break;
    case 3: asm volatile("s_waitcnt vmcnt(18)" ::: "memory"); break;
    case 4: asm volatile("s_waitcnt vmcnt(24)" ::: "memory"); break;
    default: asm volatile("s_waitcnt vmcnt(30)" ::: "memory"); break;
  }
}

__global__ void __launch_bounds__(NTHREADS, 3) fused_a2a_gemm(Params p) {
  extern __shared__ __attribute__((aligned(16))) unsigned char smem_generic[];
  lds_u8* smem = (lds_u8*)smem_generic;
  const int tid = threadIdx.x;
  const int wave = __builtin_amdgcn_readfirstlane(tid >> 6);
  const int lane = tid & 63;
  const int b = blockIdx.x;
  const int rank = p.rank;
  const int xcd = b & 7, idx = b >> 3;
  const int ntile = xcd * 4 + (idx & 3);
  const int mb = idx >> 2;
  unsigned char* myws = p.ws[rank];
  unsigned* cnt_ptr = (unsigned*)(myws + WS_CNT + (size_t)b * 64);
#define READY_AT(i) (*(volatile lds_u32*)(smem + C_READY + ((i) & 1) * 4))
#define RK0 4

  if (wave < 4) {
    const int w = wave;
    const int g = lane >> 4, i16 = lane & 15;
    unsigned rdA0, rdA1, rdB0, rdB1;
    {
      int sw = (i16 >> 1) & 7;
      rdA0 = i16 * 128 + ((g ^ sw) * 16);
      rdA1 = i16 * 128 + (((4 + g) ^ sw) * 16);
      int k = 16 * g + (i16 >> 1);
      int f = ((k >> 1) & 3) | ((k >> 2) & 4);
      rdB0 = SLOT_A + k * 128 + (((2 * w) ^ f) * 16) + (i16 & 1) * 8;
      rdB1 = SLOT_A + k * 128 + (((2 * w + 1) ^ f) * 16) + (i16 & 1) * 8;
    }
    {
      int kg;
      if (idx < 4) kg = rank * KC + idx * BK;
      else { int s_ = (idx - 4) / 7, t_ = (idx - 4) - 7 * s_; kg = ((rank + 1 + t_) & 7) * KC + s_ * BK; }
      const unsigned char* wp = p.w + (size_t)(kg + w * 32 + (lane >> 5)) * NTOT + xcd * 512 + (lane & 31) * 16;
#pragma unroll
      for (int i = 0; i < 16; ++i)
        __builtin_amdgcn_global_load_lds((const unsigned*)(wp + (size_t)i * 2 * NTOT), (lds_u32*)(smem + C_JUNK), 16, 0, 0);
    }
    v4f acc[4][2];
#pragma unroll
    for (int mt = 0; mt < 4; ++mt)
#pragma unroll
      for (int nt = 0; nt < 2; ++nt) acc[mt][nt] = v4f{0.f, 0.f, 0.f, 0.f};
    v2i pb[8], qb[8];
    v4i pa[8], qa[8];
#pragma unroll
    for (int i = 0; i < 8; ++i) { pb[i] = v2i{0, 0}; qb[i] = v2i{0, 0}; pa[i] = v4i{0, 0, 0, 0}; qa[i] = v4i{0, 0, 0, 0}; }
#define TR8(dst, ad, o) asm volatile("ds_read_b64_tr_b8 %0, %1 offset:" #o : "=v"(dst) : "v"(ad))
#define RD128(dst, ad, o) asm volatile("ds_read_b128 %0, %1 offset:" #o : "=v"(dst) : "v"(ad))
#define MK8(x, y) v8i{x[0], x[1], x[2], x[3], y[0], y[1], y[2], y[3]}
#define MKB(p0, p1, p2, p3) v8i{p0[0], p0[1], p1[0], p1[1], p2[0], p2[1], p3[0], p3[1]}
#define MMA(d, wv, xv) d = __builtin_amdgcn_mfma_scale_f32_16x16x128_f8f6f4(wv, xv, d, 1, 0, 0, 0x7f7f7f7f, 0, 0x7f7f7f7f)
#define MMA_ALL(B, A) do { \
      const v8i w0_ = MKB(B[0], B[1], B[2], B[3]), w1_ = MKB(B[4], B[5], B[6], B[7]); \
      _Pragma("unroll") for (int mt = 0; mt < 4; ++mt) { const v8i x_ = MK8(A[2 * mt], A[2 * mt + 1]); \
        MMA(acc[mt][0], w0_, x_); MMA(acc[mt][1], w1_, x_); } } while (0)
#define STEP(RB, RA, MB, MA) do { \
        asm volatile("ds_read_b32 %0, %1" : "=v"(rv) : "v"(aR)); \
        TR8(RB[0], aB0, 0); TR8(RB[1], aB0, 1024); TR8(RB[2], aB0, 8192); TR8(RB[3], aB0, 9216); \
        TR8(RB[4], aB1, 0); TR8(RB[5], aB1, 1024); TR8(RB[6], aB1, 8192); TR8(RB[7], aB1, 9216); \
        RD128(RA[0], aA0, 0); RD128(RA[1], aA1, 0); \
        RD128(RA[2], aA0, 2048); RD128(RA[3], aA1, 2048); \
        RD128(RA[4], aA0, 4096); RD128(RA[5], aA1, 4096); \
        RD128(RA[6], aA0, 6144); RD128(RA[7], aA1, 6144); \
        __builtin_amdgcn_sched_barrier(0); \
        MMA_ALL(MB, MA); \
        __builtin_amdgcn_sched_barrier(0); \
        asm volatile("s_waitcnt lgkmcnt(0)" \
                     : "+v"(RB[0]), "+v"(RB[1]), "+v"(RB[2]), "+v"(RB[3]), "+v"(RB[4]), "+v"(RB[5]), "+v"(RB[6]), "+v"(RB[7]), \
                       "+v"(RA[0]), "+v"(RA[1]), "+v"(RA[2]), "+v"(RA[3]), "+v"(RA[4]), "+v"(RA[5]), "+v"(RA[6]), "+v"(RA[7]), \
                       "+v"(acc[0][0]), "+v"(acc[0][1]), "+v"(acc[1][0]), "+v"(acc[1][1]), \
                       "+v"(acc[2][0]), "+v"(acc[2][1]), "+v"(acc[3][0]), "+v"(acc[3][1]), "+v"(rv)); \
      } while (0)
    int r = 0, iss = 4, it = 0, rk = RK0;
    int slot = 0;
    if (w == 0) STAMP(0);
#define PHASE(RB, RA, MB, MA) do { \
      for (;;) { \
        const bool rd = r < iss; BARRIER(); \
        iss = max(iss, min(rk, r + NS)); \
        if (rd) break; \
        rk = __builtin_amdgcn_readfirstlane(READY_AT(it)); ++it; } \
      if (w == 0) STAMP(1 + r); \
      const unsigned sb = (unsigned)(slot * SLOT); \
      const unsigned aB0 = sb + rdB0, aB1 = sb + rdB1, aA0 = sb + rdA0, aA1 = sb + rdA1; \
      const unsigned aR = C_READY + (it & 1) * 4; unsigned rv; \
      STEP(RB, RA, MB, MA); \
      if (rk < NSTAGE) rk = __builtin_amdgcn_readfirstlane(rv); ++it; \
      ++r; slot = (slot == NS - 1) ? 0 : slot + 1; } while (0)
    for (int rr = 0; rr < NSTAGE / 2; ++rr) {
      PHASE(pb, pa, qb, qa);
      PHASE(qb, qa, pb, pa);
    }
    MMA_ALL(qb, qa);
    if (w == 0) STAMP(33);
    const unsigned bad = __builtin_amdgcn_readfirstlane(*(volatile lds_u32*)(smem + C_BAD));
    float sc = p.sx[0] * p.sw[0];
    float* ob = p.out + (size_t)(mb * BM + i16) * NTOT + ntile * BN + w * 32 + g * 4;
#pragma unroll
    for (int mt = 0; mt < 4; ++mt)
#pragma unroll
      for (int nt = 0; nt < 2; ++nt) {
        v4f v = acc[mt][nt];
        v4f y;
#pragma unroll
        for (int q = 0; q < 4; ++q) y[q] = bad ? __builtin_nanf("") : fmaxf(v[q] * sc, 0.f);
        *(v4f*)(ob + (size_t)mt * 16 * NTOT + nt * 16) = y;
      }
  } else if (wave < 8) {
    const int w = wave - 4;
    unsigned offA[2], offB[4];
#pragma unroll
    for (int i = 0; i < 2; ++i) {
      int row = (2 * w + i) * 8 + (lane >> 3);
      offA[i] = row * KC + (((lane & 7) ^ ((row >> 1) & 7)) * 16);
    }
#pragma unroll
    for (int i = 0; i < 4; ++i) {
      int kr = (4 * w + i) * 8 + (lane >> 3);
      int f = ((kr >> 1) & 3) | ((kr >> 2) & 4);
      offB[i] = kr * NTOT + (((lane & 7) ^ f) * 16) + ntile * BN;
    }
    const unsigned char* own_a = p.x + (size_t)(rank * 512 + mb * BM) * KC;
    auto issue_at = [&](const unsigned char* ab, const unsigned char* bb, int slot) {
      lds_u8* la = smem + slot * SLOT + (2 * w) * 1024;
      lds_u8* lb = smem + slot * SLOT + SLOT_A + (4 * w) * 1024;
#pragma unroll
      for (int i = 0; i < 4; ++i)
        __builtin_amdgcn_global_load_lds((const unsigned*)(bb + offB[i]), (lds_u32*)(lb + i * 1024), 16, 0, 0);
#pragma unroll
      for (int i = 0; i < 2; ++i)
        __builtin_amdgcn_global_load_lds((const unsigned*)(ab + offA[i]), (lds_u32*)(la + i * 1024), 16, 0, AUX_A);
    };
#pragma unroll
    for (int j = 0; j < 4; ++j) issue_at(own_a + j * BK, p.w + (size_t)(rank * KC + j * BK) * NTOT, j);
    int ns = 0, nt = 0, nslot = 4;
    const unsigned char* stg_a = nullptr;
    int r = 0, iss = 4, it = 0, rk = RK0;
    for (;;) {
      const bool rd = r < iss;
      if (rd) wait_vm(iss - r - 1);
      BARRIER();
      const int lim = min(rk, r + NS);
      unsigned rv = 0;
      if (rk < NSTAGE) rv = READY_AT(it);
      if (it == 0) {
        unsigned call = __builtin_amdgcn_readfirstlane(*(volatile lds_u32*)(smem + C_CALL));
        stg_a = myws + WS_STAGE + (size_t)(call & 1) * STAGE_PAR + (size_t)mb * BM * KC;
      }
      while (iss < lim) {
        const int src = (rank + 1 + nt) & 7;
        issue_at(stg_a + (size_t)src * STAGE_SRC + ns * BK, p.w + (size_t)(src * KC + ns * BK) * NTOT, nslot);
        ++iss;
        nslot = (nslot == NS - 1) ? 0 : nslot + 1;
        if (++nt == 7) { nt = 0; ++ns; }
      }
      if (rk < NSTAGE) rk = __builtin_amdgcn_readfirstlane(rv);
      if (rd) ++r;
      ++it;
      if (r == NSTAGE) break;
    }
  } else if (wave == 8) {
    const int dst = b & 7, rg = b >> 3;
    const bool active = dst != rank;
    unsigned call = __hip_atomic_load(cnt_ptr, __ATOMIC_RELAXED, __HIP_MEMORY_SCOPE_AGENT) + 1u;
    call = __builtin_amdgcn_readfirstlane(call);
    const unsigned base = (call - 1u) * 4u;
    unsigned* flag = (unsigned*)(p.ws[dst] + WS_FLAGS + (size_t)(rank * 32 + rg) * FLAG_STRIDE);
    if (active) {
      const unsigned char* src = p.x + (size_t)(dst * 512 + rg * 16) * KC;
      unsigned char* dp = p.ws[dst] + WS_STAGE + (size_t)(call & 1) * STAGE_PAR + (size_t)rank * STAGE_SRC + (size_t)rg * 16 * KC;
      v4i d[8];
      unsigned off[8];
#pragma unroll
      for (int j = 0; j < 8; ++j) {
        off[j] = ((j & 1) * 8 + (lane >> 3)) * KC + (j >> 1) * BK + (lane & 7) * 16;
        d[j] = *(const v4i*)(src + off[j]);
      }
      asm volatile("" : "+v"(d[0]), "+v"(d[1]), "+v"(d[2]), "+v"(d[3]), "+v"(d[4]), "+v"(d[5]), "+v"(d[6]), "+v"(d[7]));
      asm volatile("s_waitcnt vmcnt(0)" ::: "memory");
#pragma unroll
      for (int j = 0; j < 8; ++j)
        asm volatile("global_store_dwordx4 %0, %1, off sc0 sc1\n\ts_nop 1" ::"v"(dp + off[j]), "v"(d[j]) : "memory");
    }
    int sent = active ? 0 : 4;
    int nissued = 8;
    int r = 0, iss = 4, it = 0, rk = RK0;
    for (;;) {
      const bool rd = r < iss;
      if (sent < 4) {
        int done = nissued - (int)vmcnt_now();
        int lvl = min(done >> 1, 4);
        if (lvl > sent) {
          if (lane == 0) __hip_atomic_store(flag, base + lvl, __ATOMIC_RELAXED, __HIP_MEMORY_SCOPE_SYSTEM);
          sent = lvl;
          ++nissued;
        }
      }
      BARRIER();
      iss = max(iss, min(rk, r + NS));
      if (rk < NSTAGE) rk = __builtin_amdgcn_readfirstlane(READY_AT(it));
      if (rd) ++r;
      ++it;
      if (r == NSTAGE) break;
    }
    if (sent < 4) {
      asm volatile("s_waitcnt vmcnt(0)" ::: "memory");
      if (lane == 0) __hip_atomic_store(flag, base + 4u, __ATOMIC_RELAXED, __HIP_MEMORY_SCOPE_SYSTEM);
    }
  } else {
    unsigned call = __hip_atomic_load(cnt_ptr, __ATOMIC_RELAXED, __HIP_MEMORY_SCOPE_AGENT) + 1u;
    call = __builtin_amdgcn_readfirstlane(call);
    const unsigned base = (call - 1u) * 4u;
    if (lane == 0) {
      *(volatile lds_u32*)(smem + C_CALL) = call;
      *(volatile lds_u32*)(smem + C_BAD) = 0u;
    }
    const unsigned char* fp = myws + WS_FLAGS + (size_t)(((lane >> 2) & 7) * 32 + mb * 4 + (lane & 3)) * FLAG_STRIDE;
    const int st_s = lane / 7, st_src = (rank + 1 + (lane - 7 * st_s)) & 7;
    const unsigned long long t0 = __builtin_amdgcn_s_memrealtime();
    int rdy = 4;
    bool pending = false;
    int r = 0, iss = 4, it = 0, rk = RK0;
    for (;;) {
      const bool rd = r < iss;
      if (rdy < NSTAGE) {
        if (pending && vmcnt_now() == 0) {
          asm volatile("s_waitcnt vmcnt(0)" ::: "memory");
          int v = (int)(*(volatile lds_u32*)(smem + C_FLAGS + lane * 4) - base);
          v = min(v, __shfl_xor(v, 1));
          v = min(v, __shfl_xor(v, 2));
          int lv = __shfl(v, st_src * 4);
          unsigned long long ok = __ballot(lane < 28 && lv > st_s);
          int nrdy = 4 + __builtin_ctzll(~ok);
          rdy = nrdy;
          pending = false;
          if (rdy < NSTAGE && __builtin_amdgcn_s_memrealtime() - t0 > DEADLINE_TICKS) {
            rdy = NSTAGE;
            if (lane == 0) {
              *(volatile lds_u32*)(smem + C_BAD) = 1u;
              __hip_atomic_store((unsigned*)(myws + WS_TMO), call, __ATOMIC_RELAXED, __HIP_MEMORY_SCOPE_AGENT);
            }
          }
        }
        if (!pending && rdy < NSTAGE) {
          __builtin_amdgcn_global_load_lds((const unsigned*)fp, (lds_u32*)(smem + C_FLAGS), 4, 0, 17);
          pending = true;
        }
      }
      if (lane == 0) READY_AT(it) = (unsigned)rdy;
      asm volatile("s_waitcnt lgkmcnt(0)" ::: "memory");
      BARRIER();
      iss = max(iss, min(rk, r + NS));
      rk = rdy;
      if (rd) ++r;
      ++it;
      if (r == NSTAGE) break;
    }
    if (lane == 0) __hip_atomic_store(cnt_ptr, call, __ATOMIC_RELAXED, __HIP_MEMORY_SCOPE_AGENT);
  }
}
}

extern "C" void kernel_launch(int rank, int nranks, ncclComm_t comm, hipStream_t stream,
                              void* const* d_in, const int* in_sizes, int n_in,
                              void* d_out, int out_size, const int* in_dtypes, int out_dtype,
                              void* const* d_out_all, void* const* d_ws_all, size_t ws_bytes) {
  if (nranks != NR || ws_bytes < WS_TOTAL) {
    fprintf(stderr, "unsupported configuration\n");
    return;
  }
  static bool attr_set = false;
  if (!attr_set) {
    (void)hipFuncSetAttribute((const void*)fused_a2a_gemm, hipFuncAttributeMaxDynamicSharedMemorySize, LDS_BYTES);
    attr_set = true;
  }
  Params p;
  p.x = (const unsigned char*)d_in[0];
  p.w = (const unsigned char*)d_in[1];
  p.sx = (const float*)d_in[2];
  p.sw = (const float*)d_in[3];
  p.out = (float*)d_out;
  for (int i = 0; i < NR; ++i) p.ws[i] = (unsigned char*)d_ws_all[i];
  p.rank = rank;
  hipLaunchKernelGGL(fused_a2a_gemm, dim3(NWG), dim3(NTHREADS), LDS_BYTES, stream, p);
}
```

```cpp
#include <hip/hip_runtime.h>
#include <rccl/rccl.h>
#include <cstdio>
#include <cstdint>

typedef int v8i __attribute__((ext_vector_type(8)));
typedef int v4i __attribute__((ext_vector_type(4)));
typedef int v2i __attribute__((ext_vector_type(2)));
typedef float v4f __attribute__((ext_vector_type(4)));
typedef __attribute__((address_space(3))) unsigned char lds_u8;
typedef __attribute__((address_space(3))) unsigned lds_u32;

#define SLOT_TICKS 24
#define AUX_A 17
namespace {
constexpr int NR = 8;
constexpr int KC = 512;
constexpr int NTOT = 4096;
constexpr int BM = 64, BN = 128, BK = 128;
constexpr int NSTAGE = 32;
constexpr int NS = 6;
constexpr int PIECE = BM * BK;
#define IPADV 1040
constexpr int IPAD = IPADV;
constexpr int SLOT_A = 8 * IPAD, SLOT_B = 16 * IPAD, SLOT = SLOT_A + SLOT_B;
constexpr int CTRL = NS * SLOT;
constexpr int C_Q = CTRL;
constexpr int C_READY = CTRL + 128;
constexpr int C_CALL = CTRL + 136;
constexpr int C_BAD = CTRL + 140;
constexpr int C_FLAGS = CTRL + 256;
constexpr int C_JUNK = CTRL + 512;
constexpr int LDS_BYTES = CTRL + 512 + 1024;
constexpr int NWG = 256;
constexpr int NTHREADS = 640;

constexpr size_t WS_STAGE = 0;
constexpr size_t STAGE_SRC = 512 * 512;
constexpr size_t STAGE_PAR = NR * STAGE_SRC;
constexpr size_t WS_FLAGS = 2 * STAGE_PAR;
constexpr size_t WS_CNT = WS_FLAGS + 4096;
constexpr size_t WS_TMO = WS_CNT + NWG * 64;
constexpr size_t WS_DBG = WS_TMO + 128;
constexpr size_t WS_TOTAL = WS_DBG + 256 * 64 * 8;
#define BARRIER() __builtin_amdgcn_s_barrier()
#define STAMP(i) do {} while (0)
constexpr unsigned long long DEADLINE_TICKS = 50000000ull;

struct Params {
  const unsigned char* x;
  const unsigned char* w;
  const float* sx;
  const float* sw;
  float* out;
  unsigned char* ws[NR];
  int rank;
};

__device__ __forceinline__ unsigned vmcnt_now() {
  unsigned s = __builtin_amdgcn_s_getreg(7 | (0 << 6) | (31 << 11));
  return (s & 15u) | ((s >> 18) & 0x30u);
}

__device__ __forceinline__ void wait_vm(int n) {
  switch (n) {
    case 0: asm volatile("s_waitcnt vmcnt(0)" ::: "memory"); break;
    case 1: asm volatile("s_waitcnt vmcnt(6)" ::: "memory"); break;
    case 2: asm volatile("s_waitcnt vmcnt(12)" ::: "memory"); break;
    case 3: asm volatile("s_waitcnt vmcnt(18)" ::: "memory"); break;
    case 4: asm volatile("s_waitcnt vmcnt(24)" ::: "memory"); break;
    default: asm volatile("s_waitcnt vmcnt(30)" ::: "memory"); break;
  }
}

__device__ __forceinline__ int lds_read_now(unsigned addr) {
  int v;
  asm volatile("ds_read_b32 %0, %1\n\ts_waitcnt lgkmcnt(0)" : "=v"(v) : "v"(addr) : "memory");
  return v;
}

__global__ void __launch_bounds__(NTHREADS, 3) fused_a2a_gemm(Params p) {
  extern __shared__ __attribute__((aligned(16))) unsigned char smem_generic[];
  lds_u8* smem = (lds_u8*)smem_generic;
  const int tid = threadIdx.x;
  const int wave = __builtin_amdgcn_readfirstlane(tid >> 6);
  const int lane = tid & 63;
  const int b = blockIdx.x;
  const int rank = p.rank;
  const int xcd = b & 7, idx = b >> 3;
  const int ntile = xcd * 4 + (idx & 3);
  const int mb = idx >> 2;
  unsigned char* myws = p.ws[rank];
  unsigned* cnt_ptr = (unsigned*)(myws + WS_CNT + (size_t)b * 64);
#define RK0 4

  if (wave < 4) {
    const int w = wave;
    const int g = lane >> 4, i16 = lane & 15;
    const unsigned rdA = (i16 & 7) * IPAD + (i16 >> 3) * 128 + g * 16;
    const unsigned rdB = SLOT_A + (i16 >> 1) * IPAD + g * 128 + w * 32 + (i16 & 1) * 8;
    v4f acc[4][2];
#pragma unroll
    for (int mt = 0; mt < 4; ++mt)
#pragma unroll
      for (int nt = 0; nt < 2; ++nt) acc[mt][nt] = v4f{0.f, 0.f, 0.f, 0.f};
    v2i pb[8], qb[8];
    v4i pa[8], qa[8];
#pragma unroll
    for (int i = 0; i < 8; ++i) { pb[i] = v2i{0, 0}; qb[i] = v2i{0, 0}; pa[i] = v4i{0, 0, 0, 0}; qa[i] = v4i{0, 0, 0, 0}; }
#define TR8(dst, ad, o) asm volatile("ds_read_b64_tr_b8 %0, %1 offset:" #o : "=v"(dst) : "v"(ad))
#define RD128(dst, ad, o) asm volatile("ds_read_b128 %0, %1 offset:" #o : "=v"(dst) : "v"(ad))
#define MK8(x, y) v8i{x[0], x[1], x[2], x[3], y[0], y[1], y[2], y[3]}
#define MKB(p0, p1, p2, p3) v8i{p0[0], p0[1], p1[0], p1[1], p2[0], p2[1], p3[0], p3[1]}
#define MMA(d, wv, xv) d = __builtin_amdgcn_mfma_scale_f32_16x16x128_f8f6f4(wv, xv, d, 1, 0, 0, 0x7f7f7f7f, 0, 0x7f7f7f7f)
#define MMA_ALL(B, A) do { \
      const v8i w0_ = MKB(B[0], B[1], B[2], B[3]), w1_ = MKB(B[4], B[5], B[6], B[7]); \
      _Pragma("unroll") for (int mt = 0; mt < 4; ++mt) { const v8i x_ = MK8(A[2 * mt], A[2 * mt + 1]); \
        MMA(acc[mt][0], w0_, x_); MMA(acc[mt][1], w1_, x_); } } while (0)
#define SB() __builtin_amdgcn_sched_barrier(0)
#define WAITALL(RB, RA) asm volatile("s_waitcnt lgkmcnt(0)" \
                     : "+v"(RB[0]), "+v"(RB[1]), "+v"(RB[2]), "+v"(RB[3]), "+v"(RB[4]), "+v"(RB[5]), "+v"(RB[6]), "+v"(RB[7]), \
                       "+v"(RA[0]), "+v"(RA[1]), "+v"(RA[2]), "+v"(RA[3]), "+v"(RA[4]), "+v"(RA[5]), "+v"(RA[6]), "+v"(RA[7]), \
                       "+v"(acc[0][0]), "+v"(acc[0][1]), "+v"(acc[1][0]), "+v"(acc[1][1]), \
                       "+v"(acc[2][0]), "+v"(acc[2][1]), "+v"(acc[3][0]), "+v"(acc[3][1]), "+v"(rv))
#define STEP(RB, RA, MB, MA) do { \
        const v8i w0_ = MKB(MB[0], MB[1], MB[2], MB[3]), w1_ = MKB(MB[4], MB[5], MB[6], MB[7]); \
        const v8i x0_ = MK8(MA[0], MA[1]), x1_ = MK8(MA[2], MA[3]), x2_ = MK8(MA[4], MA[5]), x3_ = MK8(MA[6], MA[7]); \
        TR8(RB[0], aB, 0); TR8(RB[1], aB, 8320); TR8(RB[2], aB, 512); TR8(RB[3], aB, 8832); SB(); \
        MMA(acc[0][0], w0_, x0_); SB(); TR8(RB[4], aB, 16); TR8(RB[5], aB, 8336); TR8(RB[6], aB, 528); SB(); \
        MMA(acc[0][1], w1_, x0_); SB(); TR8(RB[7], aB, 8848); RD128(RA[0], aA, 0); RD128(RA[1], aA, 64); SB(); \
        MMA(acc[1][0], w0_, x1_); SB(); RD128(RA[2], aA, 256); RD128(RA[3], aA, 320); SB(); \
        MMA(acc[1][1], w1_, x1_); SB(); RD128(RA[4], aA, 512); RD128(RA[5], aA, 576); SB(); \
        MMA(acc[2][0], w0_, x2_); SB(); RD128(RA[6], aA, 768); RD128(RA[7], aA, 832); \
        asm volatile("ds_read_b32 %0, %1" : "=v"(rv) : "v"(aR)); SB(); \
        MMA(acc[2][1], w1_, x2_); SB(); \
        MMA(acc[3][0], w0_, x3_); SB(); \
        MMA(acc[3][1], w1_, x3_); SB(); \
        WAITALL(RB, RA); } while (0)
    int r = 0, iss = 4, it = 0, rk = RK0;
    int slot = 0;
    if (w == 0) STAMP(0);
#define PHASE(RB, RA, MB, MA) do { \
      for (;;) { \
        const bool rd = r < iss; BARRIER(); \
        iss = max(iss, min(rk, r + NS)); \
        if (rd) break; \
        rk = __builtin_amdgcn_readfirstlane(lds_read_now(C_READY + (it & 1) * 4)); ++it; } \
      if (w == 0) STAMP(1 + r); \
      const unsigned sb = (unsigned)(slot * SLOT); \
      const unsigned aB = sb + rdB, aA = sb + rdA; \
      const unsigned aR = C_READY + (it & 1) * 4; unsigned rv; \
      STEP(RB, RA, MB, MA); \
      if (rk < NSTAGE) rk = __builtin_amdgcn_readfirstlane(rv); ++it; \
      ++r; slot = (slot == NS - 1) ? 0 : slot + 1; } while (0)
    PHASE(pb, pa, qb, qa);
    {
      const int rs = (idx * 4 + w) * 28 + (lane >> 5);
      const unsigned char* wp = p.w + xcd * 512 + (lane & 31) * 16;
#pragma unroll
      for (int i = 0; i < 14; ++i)
        __builtin_amdgcn_global_load_lds((const unsigned*)(wp + (size_t)(((rank + 1) * KC + rs + 2 * i) & 4095) * NTOT),
                                         (lds_u32*)(smem + C_JUNK), 16, 0, 0);
    }
    PHASE(qb, qa, pb, pa);
    for (int rr = 1; rr < NSTAGE / 2; ++rr) {
      PHASE(pb, pa, qb, qa);
      PHASE(qb, qa, pb, pa);
    }
    MMA_ALL(qb, qa);
    if (w == 0) STAMP(33);
    asm volatile("s_waitcnt vmcnt(0)" ::: "memory");
    const unsigned bad = __builtin_amdgcn_readfirstlane(lds_read_now(C_BAD));
    float sc = p.sx[0] * p.sw[0];
    float* ob = p.out + (size_t)(mb * BM + i16) * NTOT + ntile * BN + w * 32 + g * 4;
#pragma unroll
    for (int mt = 0; mt < 4; ++mt)
#pragma unroll
      for (int nt = 0; nt < 2; ++nt) {
        v4f v = acc[mt][nt];
        v4f y;
#pragma unroll
        for (int q = 0; q < 4; ++q) y[q] = bad ? __builtin_nanf("") : fmaxf(v[q] * sc, 0.f);
        *(v4f*)(ob + (size_t)mt * 16 * NTOT + nt * 16) = y;
      }
  } else if (wave < 8) {
    const int w = wave - 4;
    const unsigned offA = (2 * w + 8 * (lane >> 3)) * KC + (lane & 7) * 16;
    const unsigned offS = 2 * w * 1024 + lane * 16;
    const unsigned offB = (4 * w + 16 * (lane >> 3)) * NTOT + (lane & 7) * 16 + ntile * BN;
#define BSTEP NTOT
    const unsigned char* own_a = p.x + (size_t)(rank * 512 + mb * BM) * KC;
#pragma unroll
    for (int j = 0; j < 4; ++j) {
      lds_u8* la = smem + j * SLOT + (2 * w) * IPAD;
      lds_u8* lb = smem + j * SLOT + SLOT_A + (4 * w) * IPAD;
      const unsigned char* bb = p.w + (size_t)(rank * KC + j * BK) * NTOT;
#pragma unroll
      for (int i = 0; i < 2; ++i)
        __builtin_amdgcn_global_load_lds((const unsigned*)(own_a + j * BK + offA + i * KC), (lds_u32*)(la + i * IPAD), 16, 0, 0);
#pragma unroll
      for (int i = 0; i < 4; ++i)
        __builtin_amdgcn_global_load_lds((const unsigned*)(bb + offB + i * BSTEP), (lds_u32*)(lb + i * IPAD), 16, 0, 0);
    }
    const unsigned aQ = C_Q + min(lane, 34) * 4;
    int nslot = 4, qv = 0;
    const unsigned char* stg_a = nullptr;
    int r = 0, iss = 4, it = 0, rk = RK0;
    for (;;) {
      const bool rd = r < iss;
      if (rd) wait_vm(iss - r - 1);
      BARRIER();
      const int lim = min(rk, r + NS);
      if (rk < NSTAGE) {
        qv = lds_read_now(aQ);
        rk = __builtin_amdgcn_readlane(qv, 32 + (it & 1));
        if (it == 0) {
          unsigned call = __builtin_amdgcn_readlane(qv, 34);
          stg_a = myws + WS_STAGE + (size_t)(call & 1) * STAGE_PAR + (size_t)mb * 4 * PIECE;
        }
      }
      while (iss < lim) {
        const int c = __builtin_amdgcn_readlane(qv, iss);
        const unsigned char* ab = stg_a + (size_t)(c >> 2) * STAGE_SRC + (c & 3) * PIECE;
        const unsigned char* bb = p.w + (size_t)(c * BK) * NTOT;
        lds_u8* la = smem + nslot * SLOT + (2 * w) * IPAD;
        lds_u8* lb = smem + nslot * SLOT + SLOT_A + (4 * w) * IPAD;
#pragma unroll
        for (int i = 0; i < 2; ++i)
          __builtin_amdgcn_global_load_lds((const unsigned*)(ab + offS + i * 1024), (lds_u32*)(la + i * IPAD), 16, 0, AUX_A);
#pragma unroll
        for (int i = 0; i < 4; ++i)
          __builtin_amdgcn_global_load_lds((const unsigned*)(bb + offB + i * BSTEP), (lds_u32*)(lb + i * IPAD), 16, 0, 0);
        ++iss;
        nslot = (nslot == NS - 1) ? 0 : nslot + 1;
      }
      if (rd) ++r;
      ++it;
      if (r == NSTAGE) break;
    }
  } else if (wave == 8) {
    const unsigned long long tst = __builtin_amdgcn_s_memrealtime();
    const int dst = b & 7, pj = b >> 3, pm = pj & 7, ps = pj >> 3;
    const bool active = dst != rank;
    unsigned call = __hip_atomic_load(cnt_ptr, __ATOMIC_RELAXED, __HIP_MEMORY_SCOPE_AGENT) + 1u;
    unsigned* flag = (unsigned*)(p.ws[dst] + WS_FLAGS + (size_t)pm * 128 + (rank * 4 + ps) * 4);
    v4i d[8];
    {
      const unsigned char* src = p.x + (size_t)(dst * 512 + pm * BM + 8 * (lane >> 3)) * KC + ps * BK + (lane & 7) * 16;
#pragma unroll
      for (int j = 0; j < 8; ++j) d[j] = *(const v4i*)(src + j * KC);
      asm volatile("" : "+v"(d[0]), "+v"(d[1]), "+v"(d[2]), "+v"(d[3]), "+v"(d[4]), "+v"(d[5]), "+v"(d[6]), "+v"(d[7]), "+v"(call));
      asm volatile("s_waitcnt vmcnt(0)" ::: "memory");
    }
    call = __builtin_amdgcn_readfirstlane(call);
    unsigned char* dp = p.ws[dst] + WS_STAGE + (size_t)(call & 1) * STAGE_PAR + (size_t)rank * STAGE_SRC + (size_t)(pm * 4 + ps) * PIECE + lane * 16;
    const unsigned char* pf_a = myws + WS_STAGE + (size_t)(call & 1) * STAGE_PAR + (size_t)mb * 4 * PIECE + (idx & 3) * 2048 + lane * 16;
    const unsigned aQ = C_Q + min(lane, 33) * 4;
    int st = active ? 0 : 2;
    int pf = 4, qv = 0;
    int r = 0, iss = 4, it = 0, rk = RK0;
    for (;;) {
      const bool rd = r < iss;
      if (st == 1 && vmcnt_now() == 0) {
        if (lane == 0) __hip_atomic_store(flag, call, __ATOMIC_RELAXED, __HIP_MEMORY_SCOPE_SYSTEM);
        st = 2;
        STAMP(36);
      }
      if (st == 0 && (int)(__builtin_amdgcn_s_memrealtime() - tst) >= pj * SLOT_TICKS) {
        STAMP(35);
        asm volatile("s_waitcnt vmcnt(0)" ::: "memory");
#pragma unroll
        for (int j = 0; j < 8; ++j)
          asm volatile("global_store_dwordx4 %0, %1, off sc0 sc1\n\ts_nop 1" ::"v"(dp + j * 1024), "v"(d[j]) : "memory");
        st = 1;
      }
      BARRIER();
      iss = max(iss, min(rk, r + NS));
      if (rk < NSTAGE) {
        qv = lds_read_now(aQ);
        rk = __builtin_amdgcn_readlane(qv, 32 + (it & 1));
      }
      if (rd) ++r;
      ++it;
      if (r == NSTAGE) break;
    }
    if (st == 0) {
#pragma unroll
      for (int j = 0; j < 8; ++j)
        asm volatile("global_store_dwordx4 %0, %1, off sc0 sc1\n\ts_nop 1" ::"v"(dp + j * 1024), "v"(d[j]) : "memory");
    }
    asm volatile("s_waitcnt vmcnt(0)" ::: "memory");
    if (st < 2 && lane == 0) __hip_atomic_store(flag, call, __ATOMIC_RELAXED, __HIP_MEMORY_SCOPE_SYSTEM);
  } else {
    unsigned call = __hip_atomic_load(cnt_ptr, __ATOMIC_RELAXED, __HIP_MEMORY_SCOPE_AGENT) + 1u;
    call = __builtin_amdgcn_readfirstlane(call);
    if (lane == 0) {
      *(volatile lds_u32*)(smem + C_CALL) = call;
      *(volatile lds_u32*)(smem + C_BAD) = 0u;
    }
    const unsigned char* fp = myws + WS_FLAGS + (size_t)mb * 128 + (lane & 31) * 4;
    const bool mine = lane < 32 && (lane >> 2) != rank;
    const unsigned long long t0 = __builtin_amdgcn_s_memrealtime();
    unsigned long long done = 0;
    int rdy = 4;
    bool pending = false;
    int r = 0, iss = 4, it = 0, rk = RK0;
    for (;;) {
      const bool rd = r < iss;
      if (rdy < NSTAGE) {
        if (pending && vmcnt_now() == 0) {
          asm volatile("s_waitcnt vmcnt(0)" ::: "memory");
          int v = lds_read_now(C_FLAGS + lane * 4);
          bool tmo = __builtin_amdgcn_s_memrealtime() - t0 > DEADLINE_TICKS;
          bool ok = mine && ((int)((unsigned)v - call) >= 0 || tmo);
          const unsigned long long nw = __ballot(ok) & ~done;
          if (nw) {
            if ((nw >> lane) & 1) {
              const int pos = rdy + __builtin_popcountll(nw & ((1ull << lane) - 1ull));
              *(volatile lds_u32*)(smem + C_Q + pos * 4) = (unsigned)lane;
            }
            rdy += __builtin_popcountll(nw);
            done |= nw;
            if (tmo && lane == 0) {
              *(volatile lds_u32*)(smem + C_BAD) = 1u;
              __hip_atomic_store((unsigned*)(myws + WS_TMO), call, __ATOMIC_RELAXED, __HIP_MEMORY_SCOPE_AGENT);
            }
          }
          pending = false;
        }
        if (!pending && rdy < NSTAGE) {
          __builtin_amdgcn_global_load_lds((const unsigned*)fp, (lds_u32*)(smem + C_FLAGS), 4, 0, 17);
          pending = true;
        }
      }
      if (lane == 0) asm volatile("ds_write_b32 %0, %1" ::"v"(C_READY + (it & 1) * 4), "v"(rdy) : "memory");
      asm volatile("s_waitcnt lgkmcnt(0)" ::: "memory");
      BARRIER();
      iss = max(iss, min(rk, r + NS));
      rk = rdy;
      if (rd) ++r;
      ++it;
      if (r == NSTAGE) break;
    }
    if (lane == 0) __hip_atomic_store(cnt_ptr, call, __ATOMIC_RELAXED, __HIP_MEMORY_SCOPE_AGENT);
  }
}
}

extern "C" void kernel_launch(int rank, int nranks, ncclComm_t comm, hipStream_t stream,
                              void* const* d_in, const int* in_sizes, int n_in,
                              void* d_out, int out_size, const int* in_dtypes, int out_dtype,
                              void* const* d_out_all, void* const* d_ws_all, size_t ws_bytes) {
  if (nranks != NR || ws_bytes < WS_TOTAL) {
    fprintf(stderr, "unsupported configuration\n");
    return;
  }
  static bool attr_set = false;
  if (!attr_set) {
    (void)hipFuncSetAttribute((const void*)fused_a2a_gemm, hipFuncAttributeMaxDynamicSharedMemorySize, LDS_BYTES);
    attr_set = true;
  }
  Params p;
  p.x = (const unsigned char*)d_in[0];
  p.w = (const unsigned char*)d_in[1];
  p.sx = (const float*)d_in[2];
  p.sw = (const float*)d_in[3];
  p.out = (float*)d_out;
  for (int i = 0; i < NR; ++i) p.ws[i] = (unsigned char*)d_ws_all[i];
  p.rank = rank;
  hipLaunchKernelGGL(fused_a2a_gemm, dim3(NWG), dim3(NTHREADS), LDS_BYTES, stream, p);
}
```

```cpp
#include <hip/hip_runtime.h>
#include <rccl/rccl.h>
#include <cstdio>
#include <cstdint>

typedef int v8i __attribute__((ext_vector_type(8)));
typedef int v4i __attribute__((ext_vector_type(4)));
typedef int v2i __attribute__((ext_vector_type(2)));
typedef float v4f __attribute__((ext_vector_type(4)));
typedef __attribute__((address_space(3))) unsigned char lds_u8;
typedef __attribute__((address_space(3))) unsigned lds_u32;

#define SLOT_TICKS 23
#define EARLY 2
#define BACKLOG 4
#define NPOLL 1
#define AUX_A 17
namespace {
constexpr int NR = 8;
constexpr int KC = 512;
constexpr int NTOT = 4096;
constexpr int BM = 64, BN = 128, BK = 128;
constexpr int NSTAGE = 32;
constexpr int NS = 6;
constexpr int PIECE = BM * BK;
#define IPADV 1040
constexpr int IPAD = IPADV;
#define IPADA 1056
constexpr int IPA = IPADA;
constexpr int SLOT_A = 8 * IPA, SLOT_B = 16 * IPAD, SLOT = SLOT_A + SLOT_B;
constexpr int CTRL = NS * SLOT;
constexpr int C_Q = CTRL;
constexpr int C_READY = CTRL + 128;
constexpr int C_CALL = CTRL + 136;
constexpr int C_BAD = CTRL + 140;
constexpr int C_FLAGS = CTRL + 256;
constexpr int C_JUNK = CTRL + 1280;
constexpr int LDS_BYTES = CTRL + 1280 + 1024;
constexpr int NWG = 256;
constexpr int NTHREADS = 640;

constexpr size_t WS_STAGE = 0;
constexpr size_t STAGE_SRC = 512 * 512;
constexpr size_t STAGE_PAR = NR * STAGE_SRC;
constexpr size_t WS_FLAGS = 2 * STAGE_PAR;
constexpr size_t WS_CNT = WS_FLAGS + 4096;
constexpr size_t WS_TMO = WS_CNT + NWG * 64;
constexpr size_t WS_DBG = WS_TMO + 128;
constexpr size_t WS_TOTAL = WS_DBG + 256 * 64 * 8;
#define BARRIER() __builtin_amdgcn_s_barrier()
#define PRE_BARRIER() __builtin_amdgcn_s_barrier()
#define STAMP(i) do {} while (0)
constexpr unsigned long long DEADLINE_TICKS = 50000000ull;

struct Params {
  const unsigned char* x;
  const unsigned char* w;
  const float* sx;
  const float* sw;
  float* out;
  unsigned char* ws[NR];
  int rank;
};

__device__ __forceinline__ unsigned vmcnt_now() {
  unsigned s = __builtin_amdgcn_s_getreg(7 | (0 << 6) | (31 << 11));
  return (s & 15u) | ((s >> 18) & 0x30u);
}

__device__ __forceinline__ void wait_vm(int n) {
  switch (n) {
    case 0: asm volatile("s_waitcnt vmcnt(0)" ::: "memory"); break;
    case 1: asm volatile("s_waitcnt vmcnt(6)" ::: "memory"); break;
    case 2: asm volatile("s_waitcnt vmcnt(12)" ::: "memory"); break;
    case 3: asm volatile("s_waitcnt vmcnt(18)" ::: "memory"); break;
    case 4: asm volatile("s_waitcnt vmcnt(24)" ::: "memory"); break;
    default: asm volatile("s_waitcnt vmcnt(30)" ::: "memory"); break;
  }
}

__device__ __forceinline__ int lds_read_now(unsigned addr) {
  int v;
  asm volatile("ds_read_b32 %0, %1\n\ts_waitcnt lgkmcnt(0)" : "=v"(v) : "v"(addr) : "memory");
  return v;
}

__global__ void __launch_bounds__(NTHREADS, 3) fused_a2a_gemm(Params p) {
  extern __shared__ __attribute__((aligned(16))) unsigned char smem_generic[];
  lds_u8* smem = (lds_u8*)smem_generic;
  const int tid = threadIdx.x;
  const int wave = __builtin_amdgcn_readfirstlane(tid >> 6);
  const int lane = tid & 63;
  const int b = blockIdx.x;
  const int rank = p.rank;
  const int xcd = b & 7, idx = b >> 3;
  const int ntile = xcd * 4 + (idx & 3);
  const int mb = idx >> 2;
  unsigned char* myws = p.ws[rank];
  unsigned* cnt_ptr = (unsigned*)(myws + WS_CNT + (size_t)b * 64);
#define RK0 4

  if (wave < 4) {
    const int w = wave;
    const int g = lane >> 4, i16 = lane & 15;
    const unsigned rdA = (i16 & 7) * IPA + (i16 >> 3) * 128 + g * 16;
    const unsigned rdB = SLOT_A + (i16 >> 1) * IPAD + g * 128 + w * 32 + (i16 & 1) * 8;
    v4f acc[4][2];
#pragma unroll
    for (int mt = 0; mt < 4; ++mt)
#pragma unroll
      for (int nt = 0; nt < 2; ++nt) acc[mt][nt] = v4f{0.f, 0.f, 0.f, 0.f};
    v2i pb[8], qb[8];
    v4i pa[8], qa[8];
#pragma unroll
    for (int i = 0; i < 8; ++i) { pb[i] = v2i{0, 0}; qb[i] = v2i{0, 0}; pa[i] = v4i{0, 0, 0, 0}; qa[i] = v4i{0, 0, 0, 0}; }
#define TR8(dst, ad, o) asm volatile("ds_read_b64_tr_b8 %0, %1 offset:" #o : "=v"(dst) : "v"(ad))
#define RD128(dst, ad, o) asm volatile("ds_read_b128 %0, %1 offset:" #o : "=v"(dst) : "v"(ad))
#define MK8(x, y) v8i{x[0], x[1], x[2], x[3], y[0], y[1], y[2], y[3]}
#define MKB(p0, p1, p2, p3) v8i{p0[0], p0[1], p1[0], p1[1], p2[0], p2[1], p3[0], p3[1]}
#define MMA(d, wv, xv) d = __builtin_amdgcn_mfma_scale_f32_16x16x128_f8f6f4(wv, xv, d, 1, 0, 0, 0x7f7f7f7f, 0, 0x7f7f7f7f)
#define MMA_ALL(B, A) do { \
      const v8i w0_ = MKB(B[0], B[1], B[2], B[3]), w1_ = MKB(B[4], B[5], B[6], B[7]); \
      _Pragma("unroll") for (int mt = 0; mt < 4; ++mt) { const v8i x_ = MK8(A[2 * mt], A[2 * mt + 1]); \
        MMA(acc[mt][0], w0_, x_); MMA(acc[mt][1], w1_, x_); } } while (0)
#define SB() __builtin_amdgcn_sched_barrier(0)
#define WAITALL(RB, RA) asm volatile("s_waitcnt lgkmcnt(0)" \
                     : "+v"(RB[0]), "+v"(RB[1]), "+v"(RB[2]), "+v"(RB[3]), "+v"(RB[4]), "+v"(RB[5]), "+v"(RB[6]), "+v"(RB[7]), \
                       "+v"(RA[0]), "+v"(RA[1]), "+v"(RA[2]), "+v"(RA[3]), "+v"(RA[4]), "+v"(RA[5]), "+v"(RA[6]), "+v"(RA[7]), \
                       "+v"(acc[0][0]), "+v"(acc[0][1]), "+v"(acc[1][0]), "+v"(acc[1][1]), \
                       "+v"(acc[2][0]), "+v"(acc[2][1]), "+v"(acc[3][0]), "+v"(acc[3][1]), "+v"(rv))
#define STEP(RB, RA, MB, MA) do { \
        const v8i w0_ = MKB(MB[0], MB[1], MB[2], MB[3]), w1_ = MKB(MB[4], MB[5], MB[6], MB[7]); \
        const v8i x0_ = MK8(MA[0], MA[1]), x1_ = MK8(MA[2], MA[3]), x2_ = MK8(MA[4], MA[5]), x3_ = MK8(MA[6], MA[7]); \
        TR8(RB[0], aB, 0); TR8(RB[1], aB, 8320); TR8(RB[2], aB, 512); TR8(RB[3], aB, 8832); SB(); \
        MMA(acc[0][0], w0_, x0_); SB(); TR8(RB[4], aB, 16); TR8(RB[5], aB, 8336); TR8(RB[6], aB, 528); SB(); \
        MMA(acc[0][1], w1_, x0_); SB(); TR8(RB[7], aB, 8848); RD128(RA[0], aA, 0); RD128(RA[1], aA, 64); SB(); \
        MMA(acc[1][0], w0_, x1_); SB(); RD128(RA[2], aA, 256); RD128(RA[3], aA, 320); SB(); \
        MMA(acc[1][1], w1_, x1_); SB(); RD128(RA[4], aA, 512); RD128(RA[5], aA, 576); SB(); \
        MMA(acc[2][0], w0_, x2_); SB(); RD128(RA[6], aA, 768); RD128(RA[7], aA, 832); \
        asm volatile("ds_read_b32 %0, %1" : "=v"(rv) : "v"(aR)); SB(); \
        MMA(acc[2][1], w1_, x2_); SB(); \
        MMA(acc[3][0], w0_, x3_); SB(); \
        MMA(acc[3][1], w1_, x3_); SB(); \
        WAITALL(RB, RA); } while (0)
    const float sc = p.sx[0] * p.sw[0];
    int r = 0, iss = 4, it = 0, rk = RK0;
    int slot = 0;
    if (w == 0) STAMP(0);
    PRE_BARRIER();
#define PHASE(RB, RA, MB, MA) do { \
      for (;;) { \
        const bool rd = r < iss; BARRIER(); \
        iss = max(iss, min(rk, r + NS)); \
        if (rd) break; \
        rk = __builtin_amdgcn_readfirstlane(lds_read_now(C_READY + (it & 1) * 4)); ++it; } \
      if (w == 0) STAMP(1 + r); \
      const unsigned sb = (unsigned)(slot * SLOT); \
      const unsigned aB = sb + rdB, aA = sb + rdA; \
      const unsigned aR = C_READY + (it & 1) * 4; unsigned rv; \
      STEP(RB, RA, MB, MA); \
      if (rk < NSTAGE) rk = __builtin_amdgcn_readfirstlane(rv); ++it; \
      ++r; slot = (slot == NS - 1) ? 0 : slot + 1; } while (0)
    PHASE(pb, pa, qb, qa);
    {
      const int rs = (idx * 4 + w) * 28 + (lane >> 5);
      const unsigned char* wp = p.w + xcd * 512 + (lane & 31) * 16;
#pragma unroll
      for (int i = 0; i < 14; ++i)
        __builtin_amdgcn_global_load_lds((const unsigned*)(wp + (size_t)(((rank + 1) * KC + rs + 2 * i) & 4095) * NTOT),
                                         (lds_u32*)(smem + C_JUNK), 16, 0, 0);
    }
    PHASE(qb, qa, pb, pa);
    for (int rr = 1; rr < NSTAGE / 2; ++rr) {
      PHASE(pb, pa, qb, qa);
      PHASE(qb, qa, pb, pa);
    }
    MMA_ALL(qb, qa);
    if (w == 0) STAMP(33);
    asm volatile("s_waitcnt vmcnt(0)" ::: "memory");
    const unsigned bad = __builtin_amdgcn_readfirstlane(lds_read_now(C_BAD));
    float* ob = p.out + (size_t)(mb * BM + i16) * NTOT + ntile * BN + w * 32 + g * 4;
#pragma unroll
    for (int mt = 0; mt < 4; ++mt)
#pragma unroll
      for (int nt = 0; nt < 2; ++nt) {
        v4f v = acc[mt][nt];
        v4f y;
#pragma unroll
        for (int q = 0; q < 4; ++q) y[q] = bad ? __builtin_nanf("") : fmaxf(v[q] * sc, 0.f);
        *(v4f*)(ob + (size_t)mt * 16 * NTOT + nt * 16) = y;
      }
  } else if (wave < 8) {
    const int w = wave - 4;
    const unsigned offA = (2 * w + 8 * (lane >> 3)) * KC + (lane & 7) * 16;
    const unsigned offS = 2 * w * 1024 + lane * 16;
    const unsigned offB = (4 * w + 16 * (lane >> 3)) * NTOT + (lane & 7) * 16 + ntile * BN;
#define BSTEP NTOT
    const unsigned char* own_a = p.x + (size_t)(rank * 512 + mb * BM) * KC;
    PRE_BARRIER();
#pragma unroll
    for (int j = 0; j < 4; ++j) {
      lds_u8* la = smem + j * SLOT + (2 * w) * IPA;
      lds_u8* lb = smem + j * SLOT + SLOT_A + (4 * w) * IPAD;
      const unsigned char* bb = p.w + (size_t)(rank * KC + j * BK) * NTOT;
#pragma unroll
      for (int i = 0; i < 2; ++i)
        __builtin_amdgcn_global_load_lds((const unsigned*)(own_a + j * BK + offA + i * KC), (lds_u32*)(la + i * IPA), 16, 0, 0);
#pragma unroll
      for (int i = 0; i < 4; ++i)
        __builtin_amdgcn_global_load_lds((const unsigned*)(bb + offB + i * BSTEP), (lds_u32*)(lb + i * IPAD), 16, 0, 0);
    }
    const unsigned aQ = C_Q + min(lane, 34) * 4;
    int nslot = 4, qv = 0;
    const unsigned char* stg_a = nullptr;
    int r = 0, iss = 4, it = 0, rk = RK0;
    for (;;) {
      const bool rd = r < iss;
      if (rd) wait_vm(iss - r - 1);
      BARRIER();
      const int lim = min(rk, r + NS);
      while (iss < lim) {
        const int c = __builtin_amdgcn_readlane(qv, iss);
        const unsigned char* ab = stg_a + (size_t)(c >> 2) * STAGE_SRC + (c & 3) * PIECE;
        const unsigned char* bb = p.w + (size_t)(c * BK) * NTOT;
        lds_u8* la = smem + nslot * SLOT + (2 * w) * IPA;
        lds_u8* lb = smem + nslot * SLOT + SLOT_A + (4 * w) * IPAD;
#pragma unroll
        for (int i = 0; i < 2; ++i)
          __builtin_amdgcn_global_load_lds((const unsigned*)(ab + offS + i * 1024), (lds_u32*)(la + i * IPA), 16, 0, AUX_A);
#pragma unroll
        for (int i = 0; i < 4; ++i)
          __builtin_amdgcn_global_load_lds((const unsigned*)(bb + offB + i * BSTEP), (lds_u32*)(lb + i * IPAD), 16, 0, 0);
        ++iss;
        nslot = (nslot == NS - 1) ? 0 : nslot + 1;
      }
      if (rk < NSTAGE) {
        qv = lds_read_now(aQ);
        rk = __builtin_amdgcn_readlane(qv, 32 + (it & 1));
        if (it == 0) {
          unsigned call = __builtin_amdgcn_readlane(qv, 34);
          stg_a = myws + WS_STAGE + (size_t)(call & 1) * STAGE_PAR + (size_t)mb * 4 * PIECE;
        }
      }
      if (rd) ++r;
      ++it;
      if (r == NSTAGE) break;
    }
  } else if (wave == 8) {
    const int dst = b & 7, pj = b >> 3, pm = (pj + rank) & 7, ps = pj >> 3;
    const bool active = dst != rank;
    unsigned call = __hip_atomic_load(cnt_ptr, __ATOMIC_RELAXED, __HIP_MEMORY_SCOPE_AGENT) + 1u;
    unsigned* flag = (unsigned*)(p.ws[dst] + WS_FLAGS + (size_t)pm * 128 + (rank * 4 + ps) * 4);
    v4i d[8];
    {
      const unsigned char* src = p.x + (size_t)(dst * 512 + pm * BM + 8 * (lane >> 3)) * KC + ps * BK + (lane & 7) * 16;
#pragma unroll
      for (int j = 0; j < 8; ++j) d[j] = *(const v4i*)(src + j * KC);
      asm volatile("" : "+v"(d[0]), "+v"(d[1]), "+v"(d[2]), "+v"(d[3]), "+v"(d[4]), "+v"(d[5]), "+v"(d[6]), "+v"(d[7]), "+v"(call));
      asm volatile("s_waitcnt vmcnt(0)" ::: "memory");
    }
    const unsigned long long tst = __builtin_amdgcn_s_memrealtime();
    call = __builtin_amdgcn_readfirstlane(call);
    unsigned char* dp = p.ws[dst] + WS_STAGE + (size_t)(call & 1) * STAGE_PAR + (size_t)rank * STAGE_SRC + (size_t)(pm * 4 + ps) * PIECE + lane * 16;
    const unsigned char* pf_a = myws + WS_STAGE + (size_t)(call & 1) * STAGE_PAR + (size_t)mb * 4 * PIECE + (idx & 3) * 2048 + lane * 16;
    const unsigned aQ = C_Q + min(lane, 33) * 4;
    int st = active ? 0 : 2;
    if (st == 0 && pj < EARLY) {
      STAMP(35);
      asm volatile("s_waitcnt vmcnt(0)" ::: "memory");
#pragma unroll
      for (int j = 0; j < 8; ++j)
        asm volatile("global_store_dwordx4 %0, %1, off sc0 sc1\n\ts_nop 1" ::"v"(dp + j * 1024), "v"(d[j]) : "memory");
      st = 1;
    }
    PRE_BARRIER();
    int pf = 4, qv = 0;
    int r = 0, iss = 4, it = 0, rk = RK0;
    for (;;) {
      const bool rd = r < iss;
      if (st == 1 && vmcnt_now() == 0) {
        if (lane == 0) __hip_atomic_store(flag, call, __ATOMIC_RELAXED, __HIP_MEMORY_SCOPE_SYSTEM);
        st = 2;
        STAMP(36);
      }
      if (st == 0 && (int)(__builtin_amdgcn_s_memrealtime() - tst) >= pj * SLOT_TICKS) {
        STAMP(35);
        asm volatile("s_waitcnt vmcnt(0)" ::: "memory");
#pragma unroll
        for (int j = 0; j < 8; ++j)
          asm volatile("global_store_dwordx4 %0, %1, off sc0 sc1\n\ts_nop 1" ::"v"(dp + j * 1024), "v"(d[j]) : "memory");
        st = 1;
      }
      BARRIER();
      iss = max(iss, min(rk, r + NS));
      if (rk < NSTAGE) {
        qv = lds_read_now(aQ);
        rk = __builtin_amdgcn_readlane(qv, 32 + (it & 1));
      }
      if (rd) ++r;
      ++it;
      if (r == NSTAGE) break;
    }
    if (st == 0) {
#pragma unroll
      for (int j = 0; j < 8; ++j)
        asm volatile("global_store_dwordx4 %0, %1, off sc0 sc1\n\ts_nop 1" ::"v"(dp + j * 1024), "v"(d[j]) : "memory");
    }
    asm volatile("s_waitcnt vmcnt(0)" ::: "memory");
    if (st < 2 && lane == 0) __hip_atomic_store(flag, call, __ATOMIC_RELAXED, __HIP_MEMORY_SCOPE_SYSTEM);
  } else {
    unsigned call = __hip_atomic_load(cnt_ptr, __ATOMIC_RELAXED, __HIP_MEMORY_SCOPE_AGENT) + 1u;
    __builtin_amdgcn_sched_barrier(0);
    PRE_BARRIER();
    __builtin_amdgcn_sched_barrier(0);
    call = __builtin_amdgcn_readfirstlane(call);
    if (lane == 0) {
      *(volatile lds_u32*)(smem + C_CALL) = call;
      *(volatile lds_u32*)(smem + C_BAD) = 0u;
    }
    const unsigned char* fp = myws + WS_FLAGS + (size_t)mb * 128 + (lane & 31) * 4;
    const bool mine = lane < 32 && (lane >> 2) != rank;
    const unsigned long long t0 = __builtin_amdgcn_s_memrealtime();
    unsigned done = 0;
    const unsigned below = (1u << (lane & 31)) - 1u;
    int rdy = 4;
    int npoll = 0, infl = 0;
    int r = 0, iss = 4, it = 0, rk = RK0;
    for (;;) {
      const bool rd = r < iss;
      if (rdy < NSTAGE) {
        const int c = (int)vmcnt_now();
        if (c < infl) {
          infl = c;
          int v = lds_read_now(C_FLAGS + ((npoll - c - 1) & 3) * 256 + lane * 4);
          bool tmo = __builtin_amdgcn_s_memrealtime() - t0 > DEADLINE_TICKS;
          bool ok = mine && ((int)((unsigned)v - call) >= 0 || tmo);
          const unsigned nw = (unsigned)__ballot(ok) & ~done;
          if (nw) {
            if (ok && ((nw >> (lane & 31)) & 1))
              asm volatile("ds_write_b32 %0, %1" ::"v"(C_Q + (rdy + __builtin_popcount(nw & below)) * 4), "v"(lane) : "memory");
            rdy += __builtin_popcount(nw);
            done |= nw;
            if (tmo && lane == 0) {
              asm volatile("ds_write_b32 %0, %1" ::"v"(C_BAD), "v"(1) : "memory");
              __hip_atomic_store((unsigned*)(myws + WS_TMO), call, __ATOMIC_RELAXED, __HIP_MEMORY_SCOPE_AGENT);
            }
          }
        }
        if (infl < NPOLL && rdy < NSTAGE && rdy < r + NS + BACKLOG) {
          __builtin_amdgcn_global_load_lds((const unsigned*)fp, (lds_u32*)(smem + C_FLAGS + (npoll & 3) * 256), 4, 0, 17);
          ++npoll; ++infl;
        }
      }
      if (lane == 0) asm volatile("ds_write_b32 %0, %1" ::"v"(C_READY + (it & 1) * 4), "v"(rdy) : "memory");
      asm volatile("s_waitcnt lgkmcnt(0)" ::: "memory");
      BARRIER();
      iss = max(iss, min(rk, r + NS));
      rk = rdy;
      if (rd) ++r;
      ++it;
      if (r == NSTAGE) break;
    }
    asm volatile("s_waitcnt vmcnt(0)" ::: "memory");
    if (lane == 0) __hip_atomic_store(cnt_ptr, call, __ATOMIC_RELAXED, __HIP_MEMORY_SCOPE_AGENT);
  }
}
}

extern "C" void kernel_launch(int rank, int nranks, ncclComm_t comm, hipStream_t stream,
                              void* const* d_in, const int* in_sizes, int n_in,
                              void* d_out, int out_size, const int* in_dtypes, int out_dtype,
                              void* const* d_out_all, void* const* d_ws_all, size_t ws_bytes) {
  if (nranks != NR || ws_bytes < WS_TOTAL) {
    fprintf(stderr, "unsupported configuration\n");
    return;
  }
  static bool attr_set = false;
  if (!attr_set) {
    (void)hipFuncSetAttribute((const void*)fused_a2a_gemm, hipFuncAttributeMaxDynamicSharedMemorySize, LDS_BYTES);
    attr_set = true;
  }
  Params p;
  p.x = (const unsigned char*)d_in[0];
  p.w = (const unsigned char*)d_in[1];
  p.sx = (const float*)d_in[2];
  p.sw = (const float*)d_in[3];
  p.out = (float*)d_out;
  for (int i = 0; i < NR; ++i) p.ws[i] = (unsigned char*)d_ws_all[i];
  p.rank = rank;
  hipLaunchKernelGGL(fused_a2a_gemm, dim3(NWG), dim3(NTHREADS), LDS_BYTES, stream, p);
}
```

```cpp
#include <hip/hip_runtime.h>
#include <rccl/rccl.h>
#include <cstdio>
#include <cstdint>

typedef int v8i __attribute__((ext_vector_type(8)));
typedef int v4i __attribute__((ext_vector_type(4)));
typedef int v2i __attribute__((ext_vector_type(2)));
typedef float v4f __attribute__((ext_vector_type(4)));
typedef __attribute__((address_space(3))) unsigned char lds_u8;
typedef __attribute__((address_space(3))) unsigned lds_u32;

#define SLOT_TICKS 23
#define EARLY 2
#define BACKLOG 4
#define NPOLL 1
#define AUX_A 17
namespace {
constexpr int NR = 8;
constexpr int KC = 512;
constexpr int NTOT = 4096;
constexpr int BM = 64, BN = 128, BK = 128;
constexpr int NSTAGE = 32;
constexpr int NS = 6;
constexpr int PIECE = BM * BK;
#define IPADV 1040
constexpr int IPAD = IPADV;
#define IPADA 1056
constexpr int IPA = IPADA;
constexpr int SLOT_A = 8 * IPA, SLOT_B = 16 * IPAD, SLOT = SLOT_A + SLOT_B;
constexpr int CTRL = NS * SLOT;
constexpr int C_Q = CTRL;
constexpr int C_READY = CTRL + 128;
constexpr int C_CALL = CTRL + 136;
constexpr int C_BAD = CTRL + 140;
constexpr int C_FLAGS = CTRL + 256;
constexpr int C_JUNK = CTRL + 1280;
constexpr int LDS_BYTES = CTRL + 1280 + 1024;
constexpr int NWG = 256;
constexpr int NTHREADS = 640;

constexpr size_t WS_STAGE = 0;
constexpr size_t STAGE_SRC = 512 * 512;
constexpr size_t STAGE_PAR = NR * STAGE_SRC;
constexpr size_t WS_FLAGS = 2 * STAGE_PAR;
constexpr size_t WS_CNT = WS_FLAGS + 4096;
constexpr size_t WS_TMO = WS_CNT + NWG * 64;
constexpr size_t WS_DBG = WS_TMO + 128;
constexpr size_t WS_TOTAL = WS_DBG + 256 * 64 * 8;
#define BARRIER() __builtin_amdgcn_s_barrier()
#define PRE_BARRIER() __builtin_amdgcn_s_barrier()
#define STAMP(i) do {} while (0)
constexpr unsigned long long DEADLINE_TICKS = 50000000ull;

struct Params {
  const unsigned char* x;
  const unsigned char* w;
  const float* sx;
  const float* sw;
  float* out;
  unsigned char* ws[NR];
  int rank;
};

__device__ __forceinline__ unsigned vmcnt_now() {
  unsigned s = __builtin_amdgcn_s_getreg(7 | (0 << 6) | (31 << 11));
  return (s & 15u) | ((s >> 18) & 0x30u);
}

__device__ __forceinline__ void wait_vm(int n) {
  switch (n) {
    case 0: asm volatile("s_waitcnt vmcnt(0)" ::: "memory"); break;
    case 1: asm volatile("s_waitcnt vmcnt(6)" ::: "memory"); break;
    case 2: asm volatile("s_waitcnt vmcnt(12)" ::: "memory"); break;
    case 3: asm volatile("s_waitcnt vmcnt(18)" ::: "memory"); break;
    case 4: asm volatile("s_waitcnt vmcnt(24)" ::: "memory"); break;
    default: asm volatile("s_waitcnt vmcnt(30)" ::: "memory"); break;
  }
}

__device__ __forceinline__ int lds_read_now(unsigned addr) {
  int v;
  asm volatile("ds_read_b32 %0, %1\n\ts_waitcnt lgkmcnt(0)" : "=v"(v) : "v"(addr) : "memory");
  return v;
}

__global__ void __launch_bounds__(NTHREADS, 3) fused_a2a_gemm(Params p) {
  extern __shared__ __attribute__((aligned(16))) unsigned char smem_generic[];
  lds_u8* smem = (lds_u8*)smem_generic;
  const int tid = threadIdx.x;
  const int wave = __builtin_amdgcn_readfirstlane(tid >> 6);
  const int lane = tid & 63;
  const int b = blockIdx.x;
  const int rank = p.rank;
  const int xcd = b & 7, idx = b >> 3;
  const int ntile = xcd * 4 + (idx & 3);
  const int mb = idx >> 2;
  unsigned char* myws = p.ws[rank];
  unsigned* cnt_ptr = (unsigned*)(myws + WS_CNT + (size_t)b * 64);
#define RK0 4

  if (wave < 4) {
    const int w = wave;
    const int g = lane >> 4, i16 = lane & 15;
    const unsigned rdA = (i16 & 7) * IPA + (i16 >> 3) * 128 + g * 16;
    const unsigned rdB = SLOT_A + (i16 >> 1) * IPAD + g * 128 + w * 32 + (i16 & 1) * 8;
    v4f acc[4][2];
#pragma unroll
    for (int mt = 0; mt < 4; ++mt)
#pragma unroll
      for (int nt = 0; nt < 2; ++nt) acc[mt][nt] = v4f{0.f, 0.f, 0.f, 0.f};
    v2i pb[8], qb[8];
    v4i pa[8], qa[8];
#pragma unroll
    for (int i = 0; i < 8; ++i) { pb[i] = v2i{0, 0}; qb[i] = v2i{0, 0}; pa[i] = v4i{0, 0, 0, 0}; qa[i] = v4i{0, 0, 0, 0}; }
#define TR8(dst, ad, o) asm volatile("ds_read_b64_tr_b8 %0, %1 offset:" #o : "=v"(dst) : "v"(ad))
#define RD128(dst, ad, o) asm volatile("ds_read_b128 %0, %1 offset:" #o : "=v"(dst) : "v"(ad))
#define MK8(x, y) v8i{x[0], x[1], x[2], x[3], y[0], y[1], y[2], y[3]}
#define MKB(p0, p1, p2, p3) v8i{p0[0], p0[1], p1[0], p1[1], p2[0], p2[1], p3[0], p3[1]}
#define MMA(d, wv, xv) d = __builtin_amdgcn_mfma_scale_f32_16x16x128_f8f6f4(wv, xv, d, 1, 0, 0, 0x7f7f7f7f, 0, 0x7f7f7f7f)
#define MMA_ALL(B, A) do { \
      const v8i w0_ = MKB(B[0], B[1], B[2], B[3]), w1_ = MKB(B[4], B[5], B[6], B[7]); \
      _Pragma("unroll") for (int mt = 0; mt < 4; ++mt) { const v8i x_ = MK8(A[2 * mt], A[2 * mt + 1]); \
        MMA(acc[mt][0], w0_, x_); MMA(acc[mt][1], w1_, x_); } } while (0)
#define SB() __builtin_amdgcn_sched_barrier(0)
#define WAITALL(RB, RA) asm volatile("s_waitcnt lgkmcnt(0)" \
                     : "+v"(RB[0]), "+v"(RB[1]), "+v"(RB[2]), "+v"(RB[3]), "+v"(RB[4]), "+v"(RB[5]), "+v"(RB[6]), "+v"(RB[7]), \
                       "+v"(RA[0]), "+v"(RA[1]), "+v"(RA[2]), "+v"(RA[3]), "+v"(RA[4]), "+v"(RA[5]), "+v"(RA[6]), "+v"(RA[7]), \
                       "+v"(acc[0][0]), "+v"(acc[0][1]), "+v"(acc[1][0]), "+v"(acc[1][1]), \
                       "+v"(acc[2][0]), "+v"(acc[2][1]), "+v"(acc[3][0]), "+v"(acc[3][1]), "+v"(rv))
#define STEP(RB, RA, MB, MA) do { \
        const v8i w0_ = MKB(MB[0], MB[1], MB[2], MB[3]), w1_ = MKB(MB[4], MB[5], MB[6], MB[7]); \
        const v8i x0_ = MK8(MA[0], MA[1]), x1_ = MK8(MA[2], MA[3]), x2_ = MK8(MA[4], MA[5]), x3_ = MK8(MA[6], MA[7]); \
        TR8(RB[0], aB, 0); TR8(RB[1], aB, 8320); TR8(RB[2], aB, 512); TR8(RB[3], aB, 8832); SB(); \
        MMA(acc[0][0], w0_, x0_); SB(); TR8(RB[4], aB, 16); TR8(RB[5], aB, 8336); TR8(RB[6], aB, 528); SB(); \
        MMA(acc[0][1], w1_, x0_); SB(); TR8(RB[7], aB, 8848); RD128(RA[0], aA, 0); RD128(RA[1], aA, 64); SB(); \
        MMA(acc[1][0], w0_, x1_); SB(); RD128(RA[2], aA, 256); RD128(RA[3], aA, 320); SB(); \
        MMA(acc[1][1], w1_, x1_); SB(); RD128(RA[4], aA, 512); RD128(RA[5], aA, 576); SB(); \
        MMA(acc[2][0], w0_, x2_); SB(); RD128(RA[6], aA, 768); RD128(RA[7], aA, 832); \
        asm volatile("ds_read_b32 %0, %1" : "=v"(rv) : "v"(aR)); SB(); \
        MMA(acc[2][1], w1_, x2_); SB(); \
        MMA(acc[3][0], w0_, x3_); SB(); \
        MMA(acc[3][1], w1_, x3_); SB(); \
        WAITALL(RB, RA); } while (0)
    const float sc = p.sx[0] * p.sw[0];
    int r = 0, iss = 4, it = 0, rk = RK0;
    int slot = 0;
    if (w == 0) STAMP(0);
    PRE_BARRIER();
#define PHASE(RB, RA, MB, MA) do { \
      for (;;) { \
        const bool rd = r < iss; BARRIER(); \
        iss = max(iss, min(rk, r + NS)); \
        if (rd) break; \
        rk = __builtin_amdgcn_readfirstlane(lds_read_now(C_READY + (it & 1) * 4)); ++it; } \
      if (w == 0) STAMP(1 + r); \
      const unsigned sb = (unsigned)(slot * SLOT); \
      const unsigned aB = sb + rdB, aA = sb + rdA; \
      const unsigned aR = C_READY + (it & 1) * 4; unsigned rv; \
      STEP(RB, RA, MB, MA); \
      if (rk < NSTAGE) rk = __builtin_amdgcn_readfirstlane(rv); ++it; \
      ++r; slot = (slot == NS - 1) ? 0 : slot + 1; } while (0)
    PHASE(pb, pa, qb, qa);
    PHASE(qb, qa, pb, pa);
    for (int rr = 1; rr < NSTAGE / 2; ++rr) {
      PHASE(pb, pa, qb, qa);
      PHASE(qb, qa, pb, pa);
    }
    MMA_ALL(qb, qa);
    if (w == 0) STAMP(33);
    asm volatile("s_waitcnt vmcnt(0)" ::: "memory");
    const unsigned bad = __builtin_amdgcn_readfirstlane(lds_read_now(C_BAD));
    float* ob = p.out + (size_t)(mb * BM + i16) * NTOT + ntile * BN + w * 32 + g * 4;
#pragma unroll
    for (int mt = 0; mt < 4; ++mt)
#pragma unroll
      for (int nt = 0; nt < 2; ++nt) {
        v4f v = acc[mt][nt];
        v4f y;
#pragma unroll
        for (int q = 0; q < 4; ++q) y[q] = bad ? __builtin_nanf("") : fmaxf(v[q] * sc, 0.f);
        *(v4f*)(ob + (size_t)mt * 16 * NTOT + nt * 16) = y;
      }
  } else if (wave < 8) {
    const int w = wave - 4;
    const unsigned offA = (2 * w + 8 * (lane >> 3)) * KC + (lane & 7) * 16;
    const unsigned offS = 2 * w * 1024 + lane * 16;
    const unsigned offB = (4 * w + 16 * (lane >> 3)) * NTOT + (lane & 7) * 16 + ntile * BN;
#define BSTEP NTOT
    const unsigned char* own_a = p.x + (size_t)(rank * 512 + mb * BM) * KC;
    PRE_BARRIER();
#pragma unroll
    for (int j = 0; j < 4; ++j) {
      lds_u8* la = smem + j * SLOT + (2 * w) * IPA;
      lds_u8* lb = smem + j * SLOT + SLOT_A + (4 * w) * IPAD;
      const unsigned char* bb = p.w + (size_t)(rank * KC + j * BK) * NTOT;
#pragma unroll
      for (int i = 0; i < 2; ++i)
        __builtin_amdgcn_global_load_lds((const unsigned*)(own_a + j * BK + offA + i * KC), (lds_u32*)(la + i * IPA), 16, 0, 0);
#pragma unroll
      for (int i = 0; i < 4; ++i)
        __builtin_amdgcn_global_load_lds((const unsigned*)(bb + offB + i * BSTEP), (lds_u32*)(lb + i * IPAD), 16, 0, 0);
    }
    const unsigned aQ = C_Q + min(lane, 34) * 4;
    int nslot = 4, qv = 0;
    const unsigned char* stg_a = nullptr;
    int r = 0, iss = 4, it = 0, rk = RK0;
    for (;;) {
      const bool rd = r < iss;
      if (rd) wait_vm(iss - r - 1);
      BARRIER();
      const int lim = min(rk, r + NS);
      while (iss < lim) {
        const int c = __builtin_amdgcn_readlane(qv, iss);
        const unsigned char* ab = stg_a + (size_t)(c >> 2) * STAGE_SRC + (c & 3) * PIECE;
        const unsigned char* bb = p.w + (size_t)(c * BK) * NTOT;
        lds_u8* la = smem + nslot * SLOT + (2 * w) * IPA;
        lds_u8* lb = smem + nslot * SLOT + SLOT_A + (4 * w) * IPAD;
#pragma unroll
        for (int i = 0; i < 2; ++i)
          __builtin_amdgcn_global_load_lds((const unsigned*)(ab + offS + i * 1024), (lds_u32*)(la + i * IPA), 16, 0, AUX_A);
#pragma unroll
        for (int i = 0; i < 4; ++i)
          __builtin_amdgcn_global_load_lds((const unsigned*)(bb + offB + i * BSTEP), (lds_u32*)(lb + i * IPAD), 16, 0, 0);
        ++iss;
        nslot = (nslot == NS - 1) ? 0 : nslot + 1;
      }
      if (rk < NSTAGE) {
        qv = lds_read_now(aQ);
        rk = __builtin_amdgcn_readlane(qv, 32 + (it & 1));
        if (it == 0) {
          unsigned call = __builtin_amdgcn_readlane(qv, 34);
          stg_a = myws + WS_STAGE + (size_t)(call & 1) * STAGE_PAR + (size_t)mb * 4 * PIECE;
        }
      }
      if (rd) ++r;
      ++it;
      if (r == NSTAGE) break;
    }
  } else if (wave == 8) {
    const int dst = b & 7, pj = b >> 3, pm = (pj + rank) & 7, ps = pj >> 3;
    const bool active = dst != rank;
    unsigned call = __hip_atomic_load(cnt_ptr, __ATOMIC_RELAXED, __HIP_MEMORY_SCOPE_AGENT) + 1u;
    unsigned* flag = (unsigned*)(p.ws[dst] + WS_FLAGS + (size_t)pm * 128 + (rank * 4 + ps) * 4);
    v4i d[8];
    {
      const unsigned char* src = p.x + (size_t)(dst * 512 + pm * BM + 8 * (lane >> 3)) * KC + ps * BK + (lane & 7) * 16;
#pragma unroll
      for (int j = 0; j < 8; ++j) d[j] = *(const v4i*)(src + j * KC);
      asm volatile("" : "+v"(d[0]), "+v"(d[1]), "+v"(d[2]), "+v"(d[3]), "+v"(d[4]), "+v"(d[5]), "+v"(d[6]), "+v"(d[7]), "+v"(call));
      asm volatile("s_waitcnt vmcnt(0)" ::: "memory");
    }
    const unsigned long long tst = __builtin_amdgcn_s_memrealtime();
    call = __builtin_amdgcn_readfirstlane(call);
    unsigned char* dp = p.ws[dst] + WS_STAGE + (size_t)(call & 1) * STAGE_PAR + (size_t)rank * STAGE_SRC + (size_t)(pm * 4 + ps) * PIECE + lane * 16;
    const unsigned char* pf_a = myws + WS_STAGE + (size_t)(call & 1) * STAGE_PAR + (size_t)mb * 4 * PIECE + (idx & 3) * 2048 + lane * 16;
    const unsigned aQ = C_Q + min(lane, 33) * 4;
    int st = active ? 0 : 2;
    if (st == 0 && pj < EARLY) {
      STAMP(35);
      asm volatile("s_waitcnt vmcnt(0)" ::: "memory");
#pragma unroll
      for (int j = 0; j < 8; ++j)
        asm volatile("global_store_dwordx4 %0, %1, off sc0 sc1\n\ts_nop 1" ::"v"(dp + j * 1024), "v"(d[j]) : "memory");
      st = 1;
    }
    PRE_BARRIER();
    int pf = 4, qv = 0;
    int r = 0, iss = 4, it = 0, rk = RK0;
    for (;;) {
      const bool rd = r < iss;
      if (st == 1 && vmcnt_now() == 0) {
        if (lane == 0) __hip_atomic_store(flag, call, __ATOMIC_RELAXED, __HIP_MEMORY_SCOPE_SYSTEM);
        st = 2;
        STAMP(36);
      }
      if (st == 0 && (int)(__builtin_amdgcn_s_memrealtime() - tst) >= pj * SLOT_TICKS) {
        STAMP(35);
        asm volatile("s_waitcnt vmcnt(0)" ::: "memory");
#pragma unroll
        for (int j = 0; j < 8; ++j)
          asm volatile("global_store_dwordx4 %0, %1, off sc0 sc1\n\ts_nop 1" ::"v"(dp + j * 1024), "v"(d[j]) : "memory");
        st = 1;
      }
      BARRIER();
      iss = max(iss, min(rk, r + NS));
      if (rk < NSTAGE) {
        qv = lds_read_now(aQ);
        rk = __builtin_amdgcn_readlane(qv, 32 + (it & 1));
      }
      if (rd) ++r;
      ++it;
      if (r == NSTAGE) break;
    }
    if (st == 0) {
#pragma unroll
      for (int j = 0; j < 8; ++j)
        asm volatile("global_store_dwordx4 %0, %1, off sc0 sc1\n\ts_nop 1" ::"v"(dp + j * 1024), "v"(d[j]) : "memory");
    }
    asm volatile("s_waitcnt vmcnt(0)" ::: "memory");
    if (st < 2 && lane == 0) __hip_atomic_store(flag, call, __ATOMIC_RELAXED, __HIP_MEMORY_SCOPE_SYSTEM);
  } else {
    unsigned call = __hip_atomic_load(cnt_ptr, __ATOMIC_RELAXED, __HIP_MEMORY_SCOPE_AGENT) + 1u;
    __builtin_amdgcn_sched_barrier(0);
    PRE_BARRIER();
    __builtin_amdgcn_sched_barrier(0);
    call = __builtin_amdgcn_readfirstlane(call);
    if (lane == 0) {
      *(volatile lds_u32*)(smem + C_CALL) = call;
      *(volatile lds_u32*)(smem + C_BAD) = 0u;
    }
    const unsigned char* fp = myws + WS_FLAGS + (size_t)mb * 128 + (lane & 31) * 4;
    const bool mine = lane < 32 && (lane >> 2) != rank;
    const unsigned long long t0 = __builtin_amdgcn_s_memrealtime();
    unsigned done = 0;
    const unsigned below = (1u << (lane & 31)) - 1u;
    int rdy = 4;
    int npoll = 0, infl = 0;
    int r = 0, iss = 4, it = 0, rk = RK0;
    for (;;) {
      const bool rd = r < iss;
      if (rdy < NSTAGE) {
        const int c = (int)vmcnt_now();
        if (c < infl) {
          infl = c;
          int v = lds_read_now(C_FLAGS + ((npoll - c - 1) & 3) * 256 + lane * 4);
          bool tmo = __builtin_amdgcn_s_memrealtime() - t0 > DEADLINE_TICKS;
          bool ok = mine && ((int)((unsigned)v - call) >= 0 || tmo);
          const unsigned nw = (unsigned)__ballot(ok) & ~done;
          if (nw) {
            if (ok && ((nw >> (lane & 31)) & 1))
              asm volatile("ds_write_b32 %0, %1" ::"v"(C_Q + (rdy + __builtin_popcount(nw & below)) * 4), "v"(lane) : "memory");
            rdy += __builtin_popcount(nw);
            done |= nw;
            if (tmo && lane == 0) {
              asm volatile("ds_write_b32 %0, %1" ::"v"(C_BAD), "v"(1) : "memory");
              __hip_atomic_store((unsigned*)(myws + WS_TMO), call, __ATOMIC_RELAXED, __HIP_MEMORY_SCOPE_AGENT);
            }
          }
        }
        if (infl < NPOLL && rdy < NSTAGE && rdy < r + NS + BACKLOG) {
          __builtin_amdgcn_global_load_lds((const unsigned*)fp, (lds_u32*)(smem + C_FLAGS + (npoll & 3) * 256), 4, 0, 17);
          ++npoll; ++infl;
        }
      }
      if (lane == 0) asm volatile("ds_write_b32 %0, %1" ::"v"(C_READY + (it & 1) * 4), "v"(rdy) : "memory");
      asm volatile("s_waitcnt lgkmcnt(0)" ::: "memory");
      BARRIER();
      iss = max(iss, min(rk, r + NS));
      rk = rdy;
      if (rd) ++r;
      ++it;
      if (r == NSTAGE) break;
    }
    asm volatile("s_waitcnt vmcnt(0)" ::: "memory");
    if (lane == 0) __hip_atomic_store(cnt_ptr, call, __ATOMIC_RELAXED, __HIP_MEMORY_SCOPE_AGENT);
  }
}
}

extern "C" void kernel_launch(int rank, int nranks, ncclComm_t comm, hipStream_t stream,
                              void* const* d_in, const int* in_sizes, int n_in,
                              void* d_out, int out_size, const int* in_dtypes, int out_dtype,
                              void* const* d_out_all, void* const* d_ws_all, size_t ws_bytes) {
  if (nranks != NR || ws_bytes < WS_TOTAL) {
    fprintf(stderr, "unsupported configuration\n");
    return;
  }
  static bool attr_set = false;
  if (!attr_set) {
    (void)hipFuncSetAttribute((const void*)fused_a2a_gemm, hipFuncAttributeMaxDynamicSharedMemorySize, LDS_BYTES);
    attr_set = true;
  }
  Params p;
  p.x = (const unsigned char*)d_in[0];
  p.w = (const unsigned char*)d_in[1];
  p.sx = (const float*)d_in[2];
  p.sw = (const float*)d_in[3];
  p.out = (float*)d_out;
  for (int i = 0; i < NR; ++i) p.ws[i] = (unsigned char*)d_ws_all[i];
  p.rank = rank;
  hipLaunchKernelGGL(fused_a2a_gemm, dim3(NWG), dim3(NTHREADS), LDS_BYTES, stream, p);
}
```

```cpp
#include <hip/hip_runtime.h>
#include <rccl/rccl.h>
#include <cstdio>
#include <cstdint>

typedef int v8i __attribute__((ext_vector_type(8)));
typedef int v4i __attribute__((ext_vector_type(4)));
typedef int v2i __attribute__((ext_vector_type(2)));
typedef float v4f __attribute__((ext_vector_type(4)));
typedef __attribute__((address_space(3))) unsigned char lds_u8;
typedef __attribute__((address_space(3))) unsigned lds_u32;

#define SLOT_TICKS 23
#define EARLY 4
#define SPINB 6
#define BACKLOG 32
namespace {
constexpr int NR = 8;
constexpr int KC = 512;
constexpr int NTOT = 4096;
constexpr int BM = 64, BN = 128, BK = 128;
constexpr int NSTAGE = 32;
constexpr int NS = 6;
constexpr int PIECE = BM * BK;
constexpr int IPAD = 1040;
constexpr int IPA = 1056;
constexpr int SLOT_A = 8 * IPA, SLOT_B = 16 * IPAD, SLOT = SLOT_A + SLOT_B;
constexpr int CTRL = NS * SLOT;
constexpr int C_Q = CTRL;
constexpr int C_READY = CTRL + 128;
constexpr int C_CALL = CTRL + 136;
constexpr int C_BAD = CTRL + 140;
constexpr int LDS_BYTES = CTRL + 256;
constexpr int NWG = 256;
constexpr int NTHREADS = 640;

constexpr size_t WS_STAGE = 0;
constexpr size_t STAGE_MB = 32 * 8192;
constexpr size_t STAGE_PAR = 8 * STAGE_MB;
constexpr size_t WS_FLAGS = 2 * STAGE_PAR;
constexpr size_t WS_CNT = WS_FLAGS + 4096;
constexpr size_t WS_TMO = WS_CNT + NWG * 64;
constexpr size_t WS_DBG = WS_TMO + 128;
constexpr size_t WS_TOTAL = WS_DBG + 256 * 64 * 8;
#define BARRIER() __builtin_amdgcn_s_barrier()
#define STAMP(i) do {} while (0)
constexpr unsigned long long DEADLINE_TICKS = 50000000ull;

struct Params {
  const unsigned char* x;
  const unsigned char* w;
  const float* sx;
  const float* sw;
  float* out;
  unsigned char* ws[NR];
  int rank;
};

__device__ __forceinline__ unsigned vmcnt_now() {
  unsigned s = __builtin_amdgcn_s_getreg(7 | (0 << 6) | (31 << 11));
  return (s & 15u) | ((s >> 18) & 0x30u);
}

__device__ __forceinline__ void wait_vm(int n) {
  asm volatile(
      "s_cmp_eq_u32 %0, 4\n\ts_cbranch_scc0 1f\n\ts_waitcnt vmcnt(24)\n\ts_branch 9f\n"
      "1:\n\ts_cmp_lt_i32 %0, 0\n\ts_cbranch_scc1 9f\n\t"
      "s_waitcnt vmcnt(30)\n\ts_cmp_gt_i32 %0, 4\n\ts_cbranch_scc1 9f\n\t"
      "s_waitcnt vmcnt(18)\n\ts_cmp_eq_u32 %0, 3\n\ts_cbranch_scc1 9f\n\t"
      "s_waitcnt vmcnt(12)\n\ts_cmp_eq_u32 %0, 2\n\ts_cbranch_scc1 9f\n\t"
      "s_waitcnt vmcnt(6)\n\ts_cmp_eq_u32 %0, 1\n\ts_cbranch_scc1 9f\n\t"
      "s_waitcnt vmcnt(0)\n"
      "9:" ::"s"(n) : "scc", "memory");
}

__device__ __forceinline__ int lds_read_now(unsigned addr) {
  int v;
  asm volatile("ds_read_b32 %0, %1\n\ts_waitcnt lgkmcnt(0)" : "=v"(v) : "v"(addr) : "memory");
  return v;
}

__global__ void __launch_bounds__(NTHREADS, 3) fused_a2a_gemm(Params p) {
  extern __shared__ __attribute__((aligned(16))) unsigned char smem_generic[];
  lds_u8* smem = (lds_u8*)smem_generic;
  const int tid = threadIdx.x;
  const int wave = __builtin_amdgcn_readfirstlane(tid >> 6);
  const int lane = tid & 63;
  const int b = blockIdx.x;
  const int rank = p.rank;
  const int xcd = b & 7, idx = b >> 3;
  const int ntile = xcd * 4 + (idx & 3);
  const int mb = idx >> 2;
  unsigned char* myws = p.ws[rank];
  unsigned* cnt_ptr = (unsigned*)(myws + WS_CNT + (size_t)b * 64);

  if (wave < 4) {
    const int w = wave;
    const int g = lane >> 4, i16 = lane & 15;
    const unsigned rdA = (i16 & 7) * IPA + (i16 >> 3) * 128 + g * 16;
    const unsigned rdB = SLOT_A + (i16 >> 1) * IPAD + g * 128 + w * 32 + (i16 & 1) * 8;
    v4f acc[4][2];
#pragma unroll
    for (int mt = 0; mt < 4; ++mt)
#pragma unroll
      for (int nt = 0; nt < 2; ++nt) acc[mt][nt] = v4f{0.f, 0.f, 0.f, 0.f};
    v2i pb[8], qb[8];
    v4i pa[8], qa[8];
#pragma unroll
    for (int i = 0; i < 8; ++i) { pb[i] = v2i{0, 0}; qb[i] = v2i{0, 0}; pa[i] = v4i{0, 0, 0, 0}; qa[i] = v4i{0, 0, 0, 0}; }
#define TR8(dst, ad, o) asm volatile("ds_read_b64_tr_b8 %0, %1 offset:" #o : "=v"(dst) : "v"(ad))
#define RD128(dst, ad, o) asm volatile("ds_read_b128 %0, %1 offset:" #o : "=v"(dst) : "v"(ad))
#define MK8(x, y) v8i{x[0], x[1], x[2], x[3], y[0], y[1], y[2], y[3]}
#define MKB(p0, p1, p2, p3) v8i{p0[0], p0[1], p1[0], p1[1], p2[0], p2[1], p3[0], p3[1]}
#define MMA(d, wv, xv) d = __builtin_amdgcn_mfma_scale_f32_16x16x128_f8f6f4(wv, xv, d, 1, 0, 0, 0x7f7f7f7f, 0, 0x7f7f7f7f)
#define MMA_ALL(B, A) do { \
      const v8i w0_ = MKB(B[0], B[1], B[2], B[3]), w1_ = MKB(B[4], B[5], B[6], B[7]); \
      _Pragma("unroll") for (int mt = 0; mt < 4; ++mt) { const v8i x_ = MK8(A[2 * mt], A[2 * mt + 1]); \
        MMA(acc[mt][0], w0_, x_); MMA(acc[mt][1], w1_, x_); } } while (0)
#define SB() __builtin_amdgcn_sched_barrier(0)
#define WAITALL(RB, RA) asm volatile("s_waitcnt lgkmcnt(0)" \
                     : "+v"(RB[0]), "+v"(RB[1]), "+v"(RB[2]), "+v"(RB[3]), "+v"(RB[4]), "+v"(RB[5]), "+v"(RB[6]), "+v"(RB[7]), \
                       "+v"(RA[0]), "+v"(RA[1]), "+v"(RA[2]), "+v"(RA[3]), "+v"(RA[4]), "+v"(RA[5]), "+v"(RA[6]), "+v"(RA[7]), \
                       "+v"(acc[0][0]), "+v"(acc[0][1]), "+v"(acc[1][0]), "+v"(acc[1][1]), \
                       "+v"(acc[2][0]), "+v"(acc[2][1]), "+v"(acc[3][0]), "+v"(acc[3][1]))
#define TAKE() do { iss = max(iss, min(rk, r + NS)); rk = __builtin_amdgcn_readfirstlane(rv); aR ^= 4u; } while (0)
#define STEP(RB, RA, MB, MA) do { \
        const v8i w0_ = MKB(MB[0], MB[1], MB[2], MB[3]), w1_ = MKB(MB[4], MB[5], MB[6], MB[7]); \
        const v8i x0_ = MK8(MA[0], MA[1]), x1_ = MK8(MA[2], MA[3]), x2_ = MK8(MA[4], MA[5]), x3_ = MK8(MA[6], MA[7]); \
        TR8(RB[0], aB, 0); asm volatile("ds_read_b32 %0, %1" : "=v"(rv) : "v"(aR)); \
        TR8(RB[1], aB, 8320); TR8(RB[2], aB, 512); TR8(RB[3], aB, 8832); SB(); \
        MMA(acc[0][0], w0_, x0_); SB(); TR8(RB[4], aB, 16); TR8(RB[5], aB, 8336); TR8(RB[6], aB, 528); SB(); \
        MMA(acc[0][1], w1_, x0_); SB(); TR8(RB[7], aB, 8848); RD128(RA[0], aA, 0); RD128(RA[1], aA, 64); SB(); \
        MMA(acc[1][0], w0_, x1_); SB(); RD128(RA[2], aA, 256); RD128(RA[3], aA, 320); SB(); \
        MMA(acc[1][1], w1_, x1_); SB(); RD128(RA[4], aA, 512); RD128(RA[5], aA, 576); SB(); \
        MMA(acc[2][0], w0_, x2_); SB(); RD128(RA[6], aA, 768); RD128(RA[7], aA, 832); SB(); \
        MMA(acc[2][1], w1_, x2_); SB(); \
        asm volatile("s_waitcnt lgkmcnt(15)" : "+v"(rv)); \
        TAKE(); ++r; rd = r < iss; \
        { const int dlt = (slot == NS - 1) ? -(NS - 1) * SLOT : SLOT; slot = (slot == NS - 1) ? 0 : slot + 1; aA += dlt; aB += dlt; } \
        asm volatile("" : "+v"(aA), "+v"(aB), "+v"(aR)); SB(); \
        MMA(acc[3][0], w0_, x3_); SB(); \
        MMA(acc[3][1], w1_, x3_); SB(); \
        WAITALL(RB, RA); } while (0)
    const float sc = p.sx[0] * p.sw[0];
    int r = 0, iss = 4, rk = 4;
    int slot = 0;
    unsigned aA = rdA, aB = rdB, aR = C_READY;
    bool rd = true;
    int rv;
    if (w == 0) STAMP(0);
    BARRIER();
#define PHASE(RB, RA, MB, MA) do { \
      BARRIER(); \
      while (!rd) { \
        rv = lds_read_now(aR); \
        TAKE(); rd = r < iss; \
        BARRIER(); } \
      if (w == 0) STAMP(1 + r); \
      STEP(RB, RA, MB, MA); } while (0)
    for (int rr = 0; rr < NSTAGE / 2; ++rr) {
      PHASE(pb, pa, qb, qa);
      PHASE(qb, qa, pb, pa);
    }
    MMA_ALL(qb, qa);
    if (w == 0) STAMP(33);
    const unsigned bad = __builtin_amdgcn_readfirstlane(lds_read_now(C_BAD));
    float* ob = p.out + (size_t)(mb * BM + i16) * NTOT + ntile * BN + w * 32 + g * 4;
#pragma unroll
    for (int mt = 0; mt < 4; ++mt)
#pragma unroll
      for (int nt = 0; nt < 2; ++nt) {
        v4f v = acc[mt][nt];
        v4f y;
#pragma unroll
        for (int q = 0; q < 4; ++q) y[q] = bad ? __builtin_nanf("") : fmaxf(v[q] * sc, 0.f);
        __builtin_nontemporal_store(y, (v4f*)(ob + (size_t)mt * 16 * NTOT + nt * 16));
      }
  } else if (wave < 8) {
    const int w = wave - 4;
    const unsigned offA = (2 * w + 8 * (lane >> 3)) * KC + (lane & 7) * 16;
    const unsigned offS = 2 * w * 1024 + lane * 16;
    const unsigned offB = (4 * w + 16 * (lane >> 3)) * NTOT + (lane & 7) * 16 + ntile * BN;
    const unsigned char* own_a = p.x + (size_t)(rank * 512 + mb * BM) * KC;
    BARRIER();
#pragma unroll
    for (int j = 0; j < 4; ++j) {
      lds_u8* la = smem + j * SLOT + (2 * w) * IPA;
      lds_u8* lb = smem + j * SLOT + SLOT_A + (4 * w) * IPAD;
      const unsigned char* bb = p.w + (size_t)(rank * KC + j * BK) * NTOT;
#pragma unroll
      for (int i = 0; i < 2; ++i)
        __builtin_amdgcn_global_load_lds((const unsigned*)(own_a + j * BK + offA + i * KC), (lds_u32*)(la + i * IPA), 16, 0, 0);
#pragma unroll
      for (int i = 0; i < 4; ++i)
        __builtin_amdgcn_global_load_lds((const unsigned*)(bb + offB + i * NTOT), (lds_u32*)(lb + i * IPAD), 16, 0, 0);
    }
    const unsigned aQ = C_Q + min(lane, 34) * 4;
    const unsigned oS0 = offS, oS1 = offS + 1024;
    const unsigned oB0 = offB, oB1 = offB + NTOT, oB2 = offB + 2 * NTOT, oB3 = offB + 3 * NTOT;
    const unsigned lA = 2 * w * IPA, lB = SLOT_A + 4 * w * IPAD;
    int nslot = 4, qv = 0;
    const unsigned char* stg_a = nullptr;
#define GLDS(off, base, l, pol) asm volatile("s_mov_b32 m0, %2\n\ts_nop 0\n\tglobal_load_lds_dwordx4 %0, %1" pol ::"v"(off), "s"(base), "s"(l) : "memory")
#define ISSUE(ab, bb, sl) do { \
      GLDS(oS0, ab, sl + lA, " sc0 sc1"); GLDS(oS1, ab, sl + lA + IPA, " sc0 sc1"); \
      GLDS(oB0, bb, sl + lB, ""); GLDS(oB1, bb, sl + lB + IPAD, ""); \
      GLDS(oB2, bb, sl + lB + 2 * IPAD, ""); GLDS(oB3, bb, sl + lB + 3 * IPAD, ""); } while (0)
    int r = 0, iss = 4, it = 0, rk = 4;
    for (;;) {
      const int rd = r < iss ? 1 : 0;
      const int lim = min(rk, r + NS);
      const int c0 = __builtin_amdgcn_readlane(qv, iss & 31);
      const unsigned char* ab0 = stg_a + (unsigned)c0 * PIECE;
      const unsigned char* bb0 = p.w + (unsigned)c0 * (BK * NTOT);
      const unsigned sl0 = nslot * SLOT;
      asm volatile("" ::"s"(ab0), "s"(bb0), "s"(sl0));
      wait_vm(iss - r - 1);
      __builtin_amdgcn_sched_barrier(0);
      BARRIER();
      __builtin_amdgcn_sched_barrier(0);
      int qn;
      asm volatile("ds_read_b32 %0, %1" : "=v"(qn) : "v"(aQ) : "memory");
      if (iss < lim) {
        ISSUE(ab0, bb0, sl0);
        ++iss;
        nslot = (nslot == NS - 1) ? 0 : nslot + 1;
        while (iss < lim) {
          const int c = __builtin_amdgcn_readlane(qv, iss);
          const unsigned char* ab = stg_a + (unsigned)c * PIECE;
          const unsigned char* bb = p.w + (unsigned)c * (BK * NTOT);
          const unsigned sl = nslot * SLOT;
          ISSUE(ab, bb, sl);
          ++iss;
          nslot = (nslot == NS - 1) ? 0 : nslot + 1;
        }
      }
      asm volatile("s_waitcnt lgkmcnt(0)" : "+v"(qn)::"memory");
      if (rk < NSTAGE) {
        qv = qn;
        rk = __builtin_amdgcn_readlane(qv, 32 + (it & 1));
        if (it == 0) {
          unsigned call = __builtin_amdgcn_readlane(qv, 34);
          stg_a = myws + WS_STAGE + (size_t)(call & 1) * STAGE_PAR + (size_t)mb * STAGE_MB;
        }
      }
      r += rd;
      ++it;
      if (r == NSTAGE) break;
    }
  } else if (wave == 8) {
    const int dst = b & 7, pj = b >> 3, pm = (pj + rank) & 7, ps = pj >> 3;
    const bool active = dst != rank;
    unsigned call = __hip_atomic_load(cnt_ptr, __ATOMIC_RELAXED, __HIP_MEMORY_SCOPE_AGENT) + 1u;
    unsigned* flag = (unsigned*)(p.ws[dst] + WS_FLAGS + (size_t)pm * 128 + (rank * 4 + ps) * 4);
    v4i d[8];
    {
      const unsigned char* src = p.x + (size_t)(dst * 512 + pm * BM + 8 * (lane >> 3)) * KC + ps * BK + (lane & 7) * 16;
#pragma unroll
      for (int j = 0; j < 8; ++j) d[j] = *(const v4i*)(src + j * KC);
      asm volatile("" : "+v"(d[0]), "+v"(d[1]), "+v"(d[2]), "+v"(d[3]), "+v"(d[4]), "+v"(d[5]), "+v"(d[6]), "+v"(d[7]), "+v"(call));
      asm volatile("s_waitcnt vmcnt(0)" ::: "memory");
    }
    const unsigned long long tst = __builtin_amdgcn_s_memrealtime();
    call = __builtin_amdgcn_readfirstlane(call);
    unsigned char* dp = p.ws[dst] + WS_STAGE + (size_t)(call & 1) * STAGE_PAR + (size_t)pm * STAGE_MB + (size_t)(rank * 4 + ps) * PIECE + lane * 16;
#define PUSH() do { _Pragma("unroll") for (int j = 0; j < 8; ++j) \
      asm volatile("global_store_dwordx4 %0, %1, off sc0 sc1\n\ts_nop 1" ::"v"(dp + j * 1024), "v"(d[j]) : "memory"); } while (0)
    int st = active ? 0 : 2;
    if (st == 0 && pj < EARLY) {
      STAMP(35);
      asm volatile("s_waitcnt vmcnt(0)" ::: "memory");
      PUSH();
      st = 1;
    }
    BARRIER();
    if (st == 1) {
      asm volatile("s_waitcnt vmcnt(0)" ::: "memory");
      if (lane == 0) __hip_atomic_store(flag, call, __ATOMIC_RELAXED, __HIP_MEMORY_SCOPE_SYSTEM);
      st = 2;
      STAMP(36);
    }
    int r = 0, iss = 4, it = 0, rk = 4;
    bool prd = false;
    for (;;) {
      const bool rd = r < iss;
      if (st == 1 && prd)
        for (int k = 0; k < SPINB && vmcnt_now() != 0; ++k) __builtin_amdgcn_s_sleep(1);
      if (st == 1 && vmcnt_now() == 0) {
        if (lane == 0) __hip_atomic_store(flag, call, __ATOMIC_RELAXED, __HIP_MEMORY_SCOPE_SYSTEM);
        st = 2;
        STAMP(36);
      }
      if (st == 0 && (int)(__builtin_amdgcn_s_memrealtime() - tst) >= pj * SLOT_TICKS) {
        STAMP(35);
        asm volatile("s_waitcnt vmcnt(0)" ::: "memory");
        PUSH();
        st = 1;
      }
      BARRIER();
      iss = max(iss, min(rk, r + NS));
      if (rk < NSTAGE) rk = __builtin_amdgcn_readfirstlane(lds_read_now(C_READY + (it & 1) * 4));
      prd = rd;
      if (rd) ++r;
      ++it;
      if (r == NSTAGE) break;
    }
    if (st == 0) PUSH();
    asm volatile("s_waitcnt vmcnt(0)" ::: "memory");
    if (st < 2 && lane == 0) __hip_atomic_store(flag, call, __ATOMIC_RELAXED, __HIP_MEMORY_SCOPE_SYSTEM);
  } else {
    unsigned call = __hip_atomic_load(cnt_ptr, __ATOMIC_RELAXED, __HIP_MEMORY_SCOPE_AGENT) + 1u;
    __builtin_amdgcn_sched_barrier(0);
    BARRIER();
    __builtin_amdgcn_sched_barrier(0);
    call = __builtin_amdgcn_readfirstlane(call);
    if (lane == 0) {
      *(volatile lds_u32*)(smem + C_CALL) = call;
      *(volatile lds_u32*)(smem + C_BAD) = 0u;
    }
    const unsigned char* fp = myws + WS_FLAGS + (size_t)mb * 128 + (lane & 31) * 4;
    const bool mine = lane < 32 && (lane >> 2) != rank;
    const unsigned long long t0 = __builtin_amdgcn_s_memrealtime();
    unsigned done = 0;
    const unsigned below = (1u << (lane & 31)) - 1u;
    int rdy = 4;
    bool infl = false;
    int v = 0;
    int r = 0, iss = 4, it = 0, rk = 4;
    for (;;) {
      const bool rd = r < iss;
      if (rdy < NSTAGE) {
        if (infl && vmcnt_now() == 0) {
          infl = false;
          asm volatile("s_waitcnt vmcnt(0)" : "+v"(v)::"memory");
          bool tmo = __builtin_amdgcn_s_memrealtime() - t0 > DEADLINE_TICKS;
          bool ok = mine && ((int)((unsigned)v - call) >= 0 || tmo);
          const unsigned nw = (unsigned)__ballot(ok) & ~done;
          if (nw) {
            if (ok && ((nw >> (lane & 31)) & 1))
              asm volatile("ds_write_b32 %0, %1" ::"v"(C_Q + (rdy + __builtin_popcount(nw & below)) * 4), "v"(lane) : "memory");
            rdy += __builtin_popcount(nw);
            done |= nw;
            if (tmo && lane == 0) {
              asm volatile("ds_write_b32 %0, %1" ::"v"(C_BAD), "v"(1) : "memory");
              __hip_atomic_store((unsigned*)(myws + WS_TMO), call, __ATOMIC_RELAXED, __HIP_MEMORY_SCOPE_AGENT);
            }
          }
        }
        if (!infl && rdy < NSTAGE && rdy < r + NS + BACKLOG) {
          asm volatile("global_load_dword %0, %1, off sc0 sc1" : "=v"(v) : "v"(fp) : "memory");
          infl = true;
        }
      }
      if (lane == 0) asm volatile("ds_write_b32 %0, %1" ::"v"(C_READY + (it & 1) * 4), "v"(rdy) : "memory");
      asm volatile("s_waitcnt lgkmcnt(0)" ::: "memory");
      BARRIER();
      iss = max(iss, min(rk, r + NS));
      rk = rdy;
      if (rd) ++r;
      ++it;
      if (r == NSTAGE) break;
    }
    asm volatile("s_waitcnt vmcnt(0)" ::: "memory");
    if (lane == 0) __hip_atomic_store(cnt_ptr, call, __ATOMIC_RELAXED, __HIP_MEMORY_SCOPE_AGENT);
  }
}
}

extern "C" void kernel_launch(int rank, int nranks, ncclComm_t comm, hipStream_t stream,
                              void* const* d_in, const int* in_sizes, int n_in,
                              void* d_out, int out_size, const int* in_dtypes, int out_dtype,
                              void* const* d_out_all, void* const* d_ws_all, size_t ws_bytes) {
  if (nranks != NR || ws_bytes < WS_TOTAL) {
    fprintf(stderr, "unsupported configuration\n");
    return;
  }
  static bool attr_set = false;
  if (!attr_set) {
    (void)hipFuncSetAttribute((const void*)fused_a2a_gemm, hipFuncAttributeMaxDynamicSharedMemorySize, LDS_BYTES);
    attr_set = true;
  }
  Params p;
  p.x = (const unsigned char*)d_in[0];
  p.w = (const unsigned char*)d_in[1];
  p.sx = (const float*)d_in[2];
  p.sw = (const float*)d_in[3];
  p.out = (float*)d_out;
  for (int i = 0; i < NR; ++i) p.ws[i] = (unsigned char*)d_ws_all[i];
  p.rank = rank;
  hipLaunchKernelGGL(fused_a2a_gemm, dim3(NWG), dim3(NTHREADS), LDS_BYTES, stream, p);
}
```

```cpp
#include <hip/hip_runtime.h>
#include <rccl/rccl.h>
#include <cstdio>
#include <cstdint>

typedef int v8i __attribute__((ext_vector_type(8)));
typedef int v4i __attribute__((ext_vector_type(4)));
typedef int v2i __attribute__((ext_vector_type(2)));
typedef float v4f __attribute__((ext_vector_type(4)));
typedef __attribute__((address_space(3))) unsigned char lds_u8;
typedef __attribute__((address_space(3))) unsigned lds_u32;

#define SLOT_TICKS 23
#define EARLY 4
#define XS 40
#define XA 100
#define BACKLOG 32
#define SPINB 2
namespace {
constexpr int NR = 8;
constexpr int KC = 512;
constexpr int NTOT = 4096;
constexpr int BM = 64, BN = 128, BK = 128;
constexpr int NSTAGE = 32;
constexpr int NS = 6;
constexpr int PIECE = BM * BK;
constexpr int IPAD = 1040;
constexpr int IPA = 1056;
constexpr int SLOT_A = 8 * IPA, SLOT_B = 16 * IPAD, SLOT = SLOT_A + SLOT_B;
constexpr int CTRL = NS * SLOT;
constexpr int C_Q = CTRL;
constexpr int C_READY = CTRL + 128;
constexpr int C_CALL = CTRL + 136;
constexpr int C_BAD = CTRL + 140;
constexpr int C_TR = CTRL + 256;
constexpr int LDS_BYTES = CTRL + 512;
constexpr int NWG = 256;
constexpr int NTHREADS = 640;

constexpr size_t WS_STAGE = 0;
constexpr size_t STAGE_MB = 32 * 8192;
constexpr size_t STAGE_PAR = 8 * STAGE_MB;
constexpr size_t WS_FLAGS = 2 * STAGE_PAR;
constexpr size_t WS_CNT = WS_FLAGS + 4096;
constexpr size_t WS_TMO = WS_CNT + NWG * 64;
constexpr size_t WS_DBG = WS_TMO + 128;
constexpr size_t WS_TOTAL = WS_DBG + 256 * 64 * 8;
#define BARRIER() __builtin_amdgcn_s_barrier()
#define LSTAMP(i) do {} while (0)
#define STAMP(i) do {} while (0)
constexpr unsigned long long DEADLINE_TICKS = 50000000ull;

struct Params {
  const unsigned char* x;
  const unsigned char* w;
  const float* sx;
  const float* sw;
  float* out;
  unsigned char* ws[NR];
  int rank;
};

__device__ __forceinline__ unsigned vmcnt_now() {
  unsigned s = __builtin_amdgcn_s_getreg(7 | (0 << 6) | (31 << 11));
  return (s & 15u) | ((s >> 18) & 0x30u);
}

__device__ __forceinline__ void wait_vm(int n) {
  asm volatile(
      "s_cmp_eq_u32 %0, 4\n\ts_cbranch_scc0 1f\n\ts_waitcnt vmcnt(24)\n\ts_branch 9f\n"
      "1:\n\ts_cmp_lt_i32 %0, 0\n\ts_cbranch_scc1 9f\n\t"
      "s_waitcnt vmcnt(30)\n\ts_cmp_gt_i32 %0, 4\n\ts_cbranch_scc1 9f\n\t"
      "s_waitcnt vmcnt(18)\n\ts_cmp_eq_u32 %0, 3\n\ts_cbranch_scc1 9f\n\t"
      "s_waitcnt vmcnt(12)\n\ts_cmp_eq_u32 %0, 2\n\ts_cbranch_scc1 9f\n\t"
      "s_waitcnt vmcnt(6)\n\ts_cmp_eq_u32 %0, 1\n\ts_cbranch_scc1 9f\n\t"
      "s_waitcnt vmcnt(0)\n"
      "9:" ::"s"(n) : "scc", "memory");
}

__device__ __forceinline__ int lds_read_now(unsigned addr) {
  int v;
  asm volatile("ds_read_b32 %0, %1\n\ts_waitcnt lgkmcnt(0)" : "=v"(v) : "v"(addr) : "memory");
  return v;
}

#define AHEAD 2
#define WANT() (rk < min(NSTAGE, r + NS + AHEAD))
__global__ void __launch_bounds__(NTHREADS, 3) fused_a2a_gemm(Params p) {
  extern __shared__ __attribute__((aligned(16))) unsigned char smem_generic[];
  lds_u8* smem = (lds_u8*)smem_generic;
  const int tid = threadIdx.x;
  const int wave = __builtin_amdgcn_readfirstlane(tid >> 6);
  const int lane = tid & 63;
  const int b = blockIdx.x;
  const int rank = p.rank;
  const int xcd = b & 7, idx = b >> 3;
  const int ntile = xcd * 4 + (idx & 3);
  const int mb = idx >> 2;
  unsigned char* myws = p.ws[rank];
  unsigned* cnt_ptr = (unsigned*)(myws + WS_CNT + (size_t)b * 64);

  if (wave < 4) {
    const int w = wave;
    const int g = lane >> 4, i16 = lane & 15;
    const unsigned rdA = (i16 & 7) * IPA + (i16 >> 3) * 128 + g * 16;
    const unsigned rdB = SLOT_A + (i16 >> 1) * IPAD + g * 128 + w * 32 + (i16 & 1) * 8;
    v4f acc[4][2];
#pragma unroll
    for (int mt = 0; mt < 4; ++mt)
#pragma unroll
      for (int nt = 0; nt < 2; ++nt) acc[mt][nt] = v4f{0.f, 0.f, 0.f, 0.f};
    v2i pb[8], qb[8];
    v4i pa[8], qa[8];
#pragma unroll
    for (int i = 0; i < 8; ++i) { pb[i] = v2i{0, 0}; qb[i] = v2i{0, 0}; pa[i] = v4i{0, 0, 0, 0}; qa[i] = v4i{0, 0, 0, 0}; }
#define TR8(dst, ad, o) asm volatile("ds_read_b64_tr_b8 %0, %1 offset:" #o : "=v"(dst) : "v"(ad))
#define RD128(dst, ad, o) asm volatile("ds_read_b128 %0, %1 offset:" #o : "=v"(dst) : "v"(ad))
#define MK8(x, y) v8i{x[0], x[1], x[2], x[3], y[0], y[1], y[2], y[3]}
#define MKB(p0, p1, p2, p3) v8i{p0[0], p0[1], p1[0], p1[1], p2[0], p2[1], p3[0], p3[1]}
#define MMA(d, wv, xv) d = __builtin_amdgcn_mfma_scale_f32_16x16x128_f8f6f4(wv, xv, d, 1, 0, 0, 0x7f7f7f7f, 0, 0x7f7f7f7f)
#define MMA_ALL(B, A) do { \
      const v8i w0_ = MKB(B[0], B[1], B[2], B[3]), w1_ = MKB(B[4], B[5], B[6], B[7]); \
      _Pragma("unroll") for (int mt = 0; mt < 4; ++mt) { const v8i x_ = MK8(A[2 * mt], A[2 * mt + 1]); \
        MMA(acc[mt][0], w0_, x_); MMA(acc[mt][1], w1_, x_); } } while (0)
#define SB() __builtin_amdgcn_sched_barrier(0)
#define WAITALL(RB, RA) asm volatile("s_waitcnt lgkmcnt(0)" \
                     : "+v"(RB[0]), "+v"(RB[1]), "+v"(RB[2]), "+v"(RB[3]), "+v"(RB[4]), "+v"(RB[5]), "+v"(RB[6]), "+v"(RB[7]), \
                       "+v"(RA[0]), "+v"(RA[1]), "+v"(RA[2]), "+v"(RA[3]), "+v"(RA[4]), "+v"(RA[5]), "+v"(RA[6]), "+v"(RA[7]), \
                       "+v"(acc[0][0]), "+v"(acc[0][1]), "+v"(acc[1][0]), "+v"(acc[1][1]), \
                       "+v"(acc[2][0]), "+v"(acc[2][1]), "+v"(acc[3][0]), "+v"(acc[3][1]))
#define TAKE() do { const int rn_ = __builtin_amdgcn_readfirstlane(rv); rk = WANT() ? rn_ : rk; iss = max(iss, min(rk, r + NS)); aR ^= 4u; } while (0)
#define STEP(RB, RA, MB, MA) do { \
        const v8i w0_ = MKB(MB[0], MB[1], MB[2], MB[3]), w1_ = MKB(MB[4], MB[5], MB[6], MB[7]); \
        const v8i x0_ = MK8(MA[0], MA[1]), x1_ = MK8(MA[2], MA[3]), x2_ = MK8(MA[4], MA[5]), x3_ = MK8(MA[6], MA[7]); \
        TR8(RB[0], aB, 0); asm volatile("ds_read_b32 %0, %1" : "=v"(rv) : "v"(aR)); \
        TR8(RB[1], aB, 8320); TR8(RB[2], aB, 512); TR8(RB[3], aB, 8832); SB(); \
        MMA(acc[0][0], w0_, x0_); SB(); TR8(RB[4], aB, 16); TR8(RB[5], aB, 8336); TR8(RB[6], aB, 528); SB(); \
        MMA(acc[0][1], w1_, x0_); SB(); TR8(RB[7], aB, 8848); RD128(RA[0], aA, 0); RD128(RA[1], aA, 64); SB(); \
        MMA(acc[1][0], w0_, x1_); SB(); RD128(RA[2], aA, 256); RD128(RA[3], aA, 320); SB(); \
        MMA(acc[1][1], w1_, x1_); SB(); RD128(RA[4], aA, 512); RD128(RA[5], aA, 576); SB(); \
        MMA(acc[2][0], w0_, x2_); SB(); RD128(RA[6], aA, 768); RD128(RA[7], aA, 832); SB(); \
        MMA(acc[2][1], w1_, x2_); SB(); \
        asm volatile("s_waitcnt lgkmcnt(15)" : "+v"(rv)); \
        TAKE(); ++r; rd = r < iss; \
        { const int dlt = (slot == NS - 1) ? -(NS - 1) * SLOT : SLOT; slot = (slot == NS - 1) ? 0 : slot + 1; aA += dlt; aB += dlt; } \
        asm volatile("" : "+v"(aA), "+v"(aB), "+v"(aR)); SB(); \
        MMA(acc[3][0], w0_, x3_); SB(); \
        MMA(acc[3][1], w1_, x3_); SB(); \
        WAITALL(RB, RA); } while (0)
    const float sc = p.sx[0] * p.sw[0];
    int r = 0, iss = 4, rk = 4;
    int slot = 0;
    unsigned aA = rdA, aB = rdB, aR = C_READY;
    bool rd = true;
    int rv;
    if (w == 0) STAMP(0);
    BARRIER();
#define PHASE(RB, RA, MB, MA) do { \
      BARRIER(); \
      while (!rd) { \
        rv = lds_read_now(aR); \
        TAKE(); rd = r < iss; \
        BARRIER(); } \
      if (w == 0) STAMP(1 + r); \
      STEP(RB, RA, MB, MA); } while (0)
    for (int rr = 0; rr < NSTAGE / 2; ++rr) {
      PHASE(pb, pa, qb, qa);
      PHASE(qb, qa, pb, pa);
    }
    MMA_ALL(qb, qa);
    if (w == 0) STAMP(33);
    const unsigned bad = __builtin_amdgcn_readfirstlane(lds_read_now(C_BAD));
    float* ob = p.out + (size_t)(mb * BM + i16) * NTOT + ntile * BN + w * 32 + g * 4;
#pragma unroll
    for (int mt = 0; mt < 4; ++mt)
#pragma unroll
      for (int nt = 0; nt < 2; ++nt) {
        v4f v = acc[mt][nt];
        v4f y;
#pragma unroll
        for (int q = 0; q < 4; ++q) y[q] = bad ? __builtin_nanf("") : fmaxf(v[q] * sc, 0.f);
        __builtin_nontemporal_store(y, (v4f*)(ob + (size_t)mt * 16 * NTOT + nt * 16));
      }
  } else if (wave < 8) {
    const int w = wave - 4;
    const unsigned offA = (2 * w + 8 * (lane >> 3)) * KC + (lane & 7) * 16;
    const unsigned offS = 2 * w * 1024 + lane * 16;
    const unsigned offB = (4 * w + 16 * (lane >> 3)) * NTOT + (lane & 7) * 16 + ntile * BN;
    const unsigned char* own_a = p.x + (size_t)(rank * 512 + mb * BM) * KC;
    BARRIER();
#pragma unroll
    for (int j = 0; j < 4; ++j) {
      lds_u8* la = smem + j * SLOT + (2 * w) * IPA;
      lds_u8* lb = smem + j * SLOT + SLOT_A + (4 * w) * IPAD;
      const unsigned char* bb = p.w + (size_t)(rank * KC + j * BK) * NTOT;
#pragma unroll
      for (int i = 0; i < 2; ++i)
        __builtin_amdgcn_global_load_lds((const unsigned*)(own_a + j * BK + offA + i * KC), (lds_u32*)(la + i * IPA), 16, 0, 0);
#pragma unroll
      for (int i = 0; i < 4; ++i)
        __builtin_amdgcn_global_load_lds((const unsigned*)(bb + offB + i * NTOT), (lds_u32*)(lb + i * IPAD), 16, 0, 0);
    }
    const unsigned aQ = C_Q + min(lane, 34) * 4;
    const unsigned oS0 = offS, oS1 = offS + 1024;
    const unsigned oB0 = offB, oB1 = offB + NTOT, oB2 = offB + 2 * NTOT, oB3 = offB + 3 * NTOT;
    const unsigned lA = 2 * w * IPA, lB = SLOT_A + 4 * w * IPAD;
    int nslot = 4, qv = 0;
    const unsigned char* stg_a = nullptr;
#define GLDS(off, base, l, pol) asm volatile("s_mov_b32 m0, %2\n\ts_nop 0\n\tglobal_load_lds_dwordx4 %0, %1" pol ::"v"(off), "s"(base), "s"(l) : "memory")
#define ISSUE(ab, bb, sl) do { \
      GLDS(oS0, ab, sl + lA, " sc0 sc1"); GLDS(oS1, ab, sl + lA + IPA, " sc0 sc1"); \
      GLDS(oB0, bb, sl + lB, ""); GLDS(oB1, bb, sl + lB + IPAD, ""); \
      GLDS(oB2, bb, sl + lB + 2 * IPAD, ""); GLDS(oB3, bb, sl + lB + 3 * IPAD, ""); } while (0)
    int r = 0, iss = 4, it = 0, rk = 4;
    for (;;) {
      const int rd = r < iss ? 1 : 0;
      const int lim = min(rk, r + NS);
      const int c0 = __builtin_amdgcn_readlane(qv, iss & 31);
      const unsigned char* ab0 = stg_a + (unsigned)c0 * PIECE;
      const unsigned char* bb0 = p.w + (unsigned)c0 * (BK * NTOT);
      const unsigned sl0 = nslot * SLOT;
      asm volatile("" ::"s"(ab0), "s"(bb0), "s"(sl0));
      wait_vm(iss - r - 1);
      __builtin_amdgcn_sched_barrier(0);
      BARRIER();
      __builtin_amdgcn_sched_barrier(0);
      int qn;
      asm volatile("ds_read_b32 %0, %1" : "=v"(qn) : "v"(aQ) : "memory");
      if (iss < lim) {
        ISSUE(ab0, bb0, sl0);
        if (w == 0) LSTAMP(iss);
        ++iss;
        nslot = (nslot == NS - 1) ? 0 : nslot + 1;
        while (iss < lim) {
          const int c = __builtin_amdgcn_readlane(qv, iss);
          const unsigned char* ab = stg_a + (unsigned)c * PIECE;
          const unsigned char* bb = p.w + (unsigned)c * (BK * NTOT);
          const unsigned sl = nslot * SLOT;
          ISSUE(ab, bb, sl);
          if (w == 0) LSTAMP(iss);
          ++iss;
          nslot = (nslot == NS - 1) ? 0 : nslot + 1;
        }
      }
      asm volatile("s_waitcnt lgkmcnt(0)" : "+v"(qn)::"memory");
      if (WANT()) {
        qv = qn;
        rk = __builtin_amdgcn_readlane(qv, 32 + (it & 1));
        if (it == 0) {
          unsigned call = __builtin_amdgcn_readlane(qv, 34);
          stg_a = myws + WS_STAGE + (size_t)(call & 1) * STAGE_PAR + (size_t)mb * STAGE_MB;
        }
        const int lim2 = min(rk, r + NS);
        while (iss < lim2) {
          const int c = __builtin_amdgcn_readlane(qv, iss);
          const unsigned char* ab = stg_a + (unsigned)c * PIECE;
          const unsigned char* bb = p.w + (unsigned)c * (BK * NTOT);
          const unsigned sl = nslot * SLOT;
          ISSUE(ab, bb, sl);
          if (w == 0) LSTAMP(iss);
          ++iss;
          nslot = (nslot == NS - 1) ? 0 : nslot + 1;
        }
      }
      r += rd;
      ++it;
      if (r == NSTAGE) break;
    }
  } else if (wave == 8) {
    const int dst = b & 7, pj = b >> 3, pm = (pj + rank) & 7, ps = pj >> 3;
    const bool active = dst != rank;
    unsigned call = __hip_atomic_load(cnt_ptr, __ATOMIC_RELAXED, __HIP_MEMORY_SCOPE_AGENT) + 1u;
    unsigned* flag = (unsigned*)(p.ws[dst] + WS_FLAGS + (size_t)pm * 128 + (rank * 4 + ps) * 4);
    v4i d[8];
    {
      const unsigned char* src = p.x + (size_t)(dst * 512 + pm * BM + 8 * (lane >> 3)) * KC + ps * BK + (lane & 7) * 16;
#pragma unroll
      for (int j = 0; j < 8; ++j) d[j] = *(const v4i*)(src + j * KC);
      asm volatile("" : "+v"(d[0]), "+v"(d[1]), "+v"(d[2]), "+v"(d[3]), "+v"(d[4]), "+v"(d[5]), "+v"(d[6]), "+v"(d[7]), "+v"(call));
      asm volatile("s_waitcnt vmcnt(0)" ::: "memory");
    }
    const unsigned long long tst = __builtin_amdgcn_s_memrealtime();
    call = __builtin_amdgcn_readfirstlane(call);
    unsigned char* dp = p.ws[dst] + WS_STAGE + (size_t)(call & 1) * STAGE_PAR + (size_t)pm * STAGE_MB + (size_t)(rank * 4 + ps) * PIECE + lane * 16;
#define PUSH() do { _Pragma("unroll") for (int j = 0; j < 8; ++j) \
      asm volatile("global_store_dwordx4 %0, %1, off sc0 sc1\n\ts_nop 1" ::"v"(dp + j * 1024), "v"(d[j]) : "memory"); } while (0)
    int st = active ? 0 : 2;
    if (st == 0 && pj < EARLY) {
      STAMP(35);
      asm volatile("s_waitcnt vmcnt(0)" ::: "memory");
      PUSH();
      st = 1;
    }
    BARRIER();
    if (st == 1) {
      asm volatile("s_waitcnt vmcnt(0)" ::: "memory");
      if (lane == 0) __hip_atomic_store(flag, call, __ATOMIC_RELAXED, __HIP_MEMORY_SCOPE_SYSTEM);
      st = 2;
      STAMP(36);
    }
    unsigned long long tp = 0;
    for (;;) {
      if (st == 0) {
        if ((int)(__builtin_amdgcn_s_memrealtime() - tst) >= pj * SLOT_TICKS - XS) {
          while ((int)(__builtin_amdgcn_s_memrealtime() - tst) < pj * SLOT_TICKS) __builtin_amdgcn_s_sleep(1);
          STAMP(35);
          asm volatile("s_waitcnt vmcnt(0)" ::: "memory");
          PUSH();
          tp = __builtin_amdgcn_s_memrealtime();
          st = 1;
        }
      } else {
        if (st == 1) {
          if ((int)(__builtin_amdgcn_s_memrealtime() - tp) >= XA) asm volatile("s_waitcnt vmcnt(0)" ::: "memory");
          else for (int k = 0; k < SPINB && vmcnt_now() != 0; ++k) __builtin_amdgcn_s_sleep(1);
        }
        if (vmcnt_now() == 0) {
          if (st == 2) break;
          if (lane == 0) __hip_atomic_store(flag, call, __ATOMIC_RELAXED, __HIP_MEMORY_SCOPE_SYSTEM);
          STAMP(36);
          st = 2;
        }
      }
      BARRIER();
    }
  } else {
    unsigned call = __hip_atomic_load(cnt_ptr, __ATOMIC_RELAXED, __HIP_MEMORY_SCOPE_AGENT) + 1u;
    __builtin_amdgcn_sched_barrier(0);
    BARRIER();
    __builtin_amdgcn_sched_barrier(0);
    call = __builtin_amdgcn_readfirstlane(call);
    if (lane == 0) {
      *(volatile lds_u32*)(smem + C_CALL) = call;
      *(volatile lds_u32*)(smem + C_BAD) = 0u;
    }
    const unsigned char* fp = myws + WS_FLAGS + (size_t)mb * 128 + (lane & 31) * 4;
    const bool mine = lane < 32 && (lane >> 2) != rank;
    const unsigned long long t0 = __builtin_amdgcn_s_memrealtime();
    unsigned done = 0;
    const unsigned below = (1u << (lane & 31)) - 1u;
    int rdy = 4;
    bool infl = false;
    int v = 0;
    int fin = 0, pub1 = 0, pub2 = 0;
    int r = 0, iss = 4, rk = 4;
    for (int it = 0; fin < 2; ++it) {
      const int rd = r < iss ? 1 : 0;
      if (rdy < NSTAGE) {
        if (infl && vmcnt_now() == 0) {
          infl = false;
          asm volatile("s_waitcnt vmcnt(0)" : "+v"(v)::"memory");
          bool tmo = __builtin_amdgcn_s_memrealtime() - t0 > DEADLINE_TICKS;
          bool ok = mine && ((int)((unsigned)v - call) >= 0 || tmo);
          const unsigned nw = (unsigned)__ballot(ok) & ~done;
          if (nw) {
            if (ok && ((nw >> (lane & 31)) & 1))
              asm volatile("ds_write_b32 %0, %1" ::"v"(C_Q + (rdy + __builtin_popcount(nw & below)) * 4), "v"(lane) : "memory");
            rdy += __builtin_popcount(nw);
            done |= nw;
            if (tmo && lane == 0) {
              asm volatile("ds_write_b32 %0, %1" ::"v"(C_BAD), "v"(1) : "memory");
              __hip_atomic_store((unsigned*)(myws + WS_TMO), call, __ATOMIC_RELAXED, __HIP_MEMORY_SCOPE_AGENT);
            }
          }
        }
        if (!infl && rdy < NSTAGE && rdy < r + NS + BACKLOG) {
          asm volatile("global_load_dword %0, %1, off sc0 sc1" : "=v"(v) : "v"(fp) : "memory");
          infl = true;
        }
      }
      if (rdy != pub2) {
        if (lane == 0) asm volatile("ds_write_b32 %0, %1" ::"v"(C_READY + (it & 1) * 4), "v"(rdy) : "memory");
        asm volatile("s_waitcnt lgkmcnt(0)" ::: "memory");
      }
      pub2 = pub1; pub1 = rdy;
      if (rdy == NSTAGE) {
        if (fin == 0 && lane == 0) __hip_atomic_store(cnt_ptr, call, __ATOMIC_RELAXED, __HIP_MEMORY_SCOPE_AGENT);
        if (fin == 0 || vmcnt_now() == 0) ++fin;
      }
      BARRIER();
      rk = WANT() ? rdy : rk;
      iss = max(iss, min(rk, r + NS));
      r += rd;
    }
  }
}
}

extern "C" void kernel_launch(int rank, int nranks, ncclComm_t comm, hipStream_t stream,
                              void* const* d_in, const int* in_sizes, int n_in,
                              void* d_out, int out_size, const int* in_dtypes, int out_dtype,
                              void* const* d_out_all, void* const* d_ws_all, size_t ws_bytes) {
  if (nranks != NR || ws_bytes < WS_TOTAL) {
    fprintf(stderr, "unsupported configuration\n");
    return;
  }
  static bool attr_set = false;
  if (!attr_set) {
    (void)hipFuncSetAttribute((const void*)fused_a2a_gemm, hipFuncAttributeMaxDynamicSharedMemorySize, LDS_BYTES);
    attr_set = true;
  }
  Params p;
  p.x = (const unsigned char*)d_in[0];
  p.w = (const unsigned char*)d_in[1];
  p.sx = (const float*)d_in[2];
  p.sw = (const float*)d_in[3];
  p.out = (float*)d_out;
  for (int i = 0; i < NR; ++i) p.ws[i] = (unsigned char*)d_ws_all[i];
  p.rank = rank;
  hipLaunchKernelGGL(fused_a2a_gemm, dim3(NWG), dim3(NTHREADS), LDS_BYTES, stream, p);
}
```

```cpp
#include <hip/hip_runtime.h>
#include <rccl/rccl.h>
#include <cstdio>
#include <cstdint>

typedef int v8i __attribute__((ext_vector_type(8)));
typedef int v4i __attribute__((ext_vector_type(4)));
typedef int v2i __attribute__((ext_vector_type(2)));
typedef float v4f __attribute__((ext_vector_type(4)));
typedef __attribute__((address_space(3))) unsigned char lds_u8;
typedef __attribute__((address_space(3))) unsigned lds_u32;

#define SLOT_T1 19
#define SLOT_T2 25
#define SLOT_N1 12
#define EARLY 4
#define XS 40
#define XA 70
#define XW 100
#define SPINB 2
namespace {
constexpr int NR = 8;
constexpr int KC = 512;
constexpr int NTOT = 4096;
constexpr int BM = 64, BN = 128, BK = 128;
constexpr int NSTAGE = 32;
constexpr int NS = 6;
constexpr int PIECE = BM * BK;
constexpr int IPAD = 1040;
constexpr int IPA = 1056;
constexpr int SLOT_A = 8 * IPA, SLOT_B = 16 * IPAD, SLOT = SLOT_A + SLOT_B;
constexpr int CTRL = NS * SLOT;
constexpr int C_Q = CTRL;
constexpr int C_READY = CTRL + 128;
constexpr int C_CALL = CTRL + 136;
constexpr int C_BAD = CTRL + 140;
constexpr int C_TR = CTRL + 256;
constexpr int LDS_BYTES = CTRL + 512;
constexpr int NWG = 256;
constexpr int NTHREADS = 640;

constexpr size_t WS_STAGE = 0;
constexpr size_t STAGE_MB = 32 * 8192;
constexpr size_t STAGE_PAR = 8 * STAGE_MB;
constexpr size_t WS_FLAGS = 2 * STAGE_PAR;
constexpr size_t WS_CNT = WS_FLAGS + 4096;
constexpr size_t WS_TMO = WS_CNT + NWG * 64;
constexpr size_t WS_DBG = WS_TMO + 128;
constexpr size_t WS_TOTAL = WS_DBG + 256 * 64 * 8;
#define BARRIER() __builtin_amdgcn_s_barrier()
#define STAMP(i) do {} while (0)
constexpr unsigned long long DEADLINE_TICKS = 50000000ull;

struct Params {
  const unsigned char* x;
  const unsigned char* w;
  const float* sx;
  const float* sw;
  float* out;
  unsigned char* ws[NR];
  int rank;
};

__device__ __forceinline__ bool vm_busy() {
  return (__builtin_amdgcn_s_getreg(7 | (0 << 6) | (31 << 11)) & 0xC0000Fu) != 0;
}

__device__ __forceinline__ void wait_vm(int n) {
  asm volatile(
      "s_cmp_eq_u32 %0, 4\n\ts_cbranch_scc0 1f\n\ts_waitcnt vmcnt(24)\n\ts_branch 9f\n"
      "1:\n\ts_cmp_lt_i32 %0, 0\n\ts_cbranch_scc1 9f\n\t"
      "s_waitcnt vmcnt(30)\n\ts_cmp_gt_i32 %0, 4\n\ts_cbranch_scc1 9f\n\t"
      "s_waitcnt vmcnt(18)\n\ts_cmp_eq_u32 %0, 3\n\ts_cbranch_scc1 9f\n\t"
      "s_waitcnt vmcnt(12)\n\ts_cmp_eq_u32 %0, 2\n\ts_cbranch_scc1 9f\n\t"
      "s_waitcnt vmcnt(6)\n\ts_cmp_eq_u32 %0, 1\n\ts_cbranch_scc1 9f\n\t"
      "s_waitcnt vmcnt(0)\n"
      "9:" ::"s"(n) : "scc", "memory");
}

__device__ __forceinline__ int lds_read_now(unsigned addr) {
  int v;
  asm volatile("ds_read_b32 %0, %1\n\ts_waitcnt lgkmcnt(0)" : "=v"(v) : "v"(addr) : "memory");
  return v;
}

#define AHEAD 2
#define WANT() (rk < min(NSTAGE, r + NS + AHEAD))
__global__ void __launch_bounds__(NTHREADS, 3) fused_a2a_gemm(Params p) {
  extern __shared__ __attribute__((aligned(16))) unsigned char smem_generic[];
  lds_u8* smem = (lds_u8*)smem_generic;
  const int tid = threadIdx.x;
  const int wave = __builtin_amdgcn_readfirstlane(tid >> 6);
  const int lane = tid & 63;
  const int b = blockIdx.x;
  const int rank = p.rank;
  const int xcd = b & 7, idx = b >> 3;
  const int ntile = xcd * 4 + (idx & 3), mb = idx >> 2;
  unsigned char* myws = p.ws[rank];
  unsigned* cnt_ptr = (unsigned*)(myws + WS_CNT + (size_t)b * 64);

  if (wave < 4) {
    const int w = wave;
    const int g = lane >> 4, i16 = lane & 15;
    const unsigned rdA = (i16 & 7) * IPA + (i16 >> 3) * 128 + g * 16;
    const unsigned rdB = SLOT_A + (i16 >> 1) * IPAD + g * 128 + w * 32 + (i16 & 1) * 8;
    v4f acc[4][2];
#pragma unroll
    for (int mt = 0; mt < 4; ++mt)
#pragma unroll
      for (int nt = 0; nt < 2; ++nt) acc[mt][nt] = v4f{0.f, 0.f, 0.f, 0.f};
    v2i pb[8], qb[8];
    v4i pa[8], qa[8];
#pragma unroll
    for (int i = 0; i < 8; ++i) { pb[i] = v2i{0, 0}; qb[i] = v2i{0, 0}; pa[i] = v4i{0, 0, 0, 0}; qa[i] = v4i{0, 0, 0, 0}; }
#define TR8(dst, ad, o) asm volatile("ds_read_b64_tr_b8 %0, %1 offset:" #o : "=v"(dst) : "v"(ad))
#define RD128(dst, ad, o) asm volatile("ds_read_b128 %0, %1 offset:" #o : "=v"(dst) : "v"(ad))
#define MK8(x, y) v8i{x[0], x[1], x[2], x[3], y[0], y[1], y[2], y[3]}
#define MKB(p0, p1, p2, p3) v8i{p0[0], p0[1], p1[0], p1[1], p2[0], p2[1], p3[0], p3[1]}
#define MMA(d, wv, xv) d = __builtin_amdgcn_mfma_scale_f32_16x16x128_f8f6f4(wv, xv, d, 1, 0, 0, 0x7f7f7f7f, 0, 0x7f7f7f7f)
#define MMA_ALL(B, A) do { \
      const v8i w0_ = MKB(B[0], B[1], B[2], B[3]), w1_ = MKB(B[4], B[5], B[6], B[7]); \
      _Pragma("unroll") for (int mt = 0; mt < 4; ++mt) { const v8i x_ = MK8(A[2 * mt], A[2 * mt + 1]); \
        MMA(acc[mt][0], w0_, x_); MMA(acc[mt][1], w1_, x_); } } while (0)
#define SB() __builtin_amdgcn_sched_barrier(0)

#define WAITALL(RB, RA) asm volatile("s_waitcnt lgkmcnt(0)" \
                     : "+v"(RB[0]), "+v"(RB[1]), "+v"(RB[2]), "+v"(RB[3]), "+v"(RB[4]), "+v"(RB[5]), "+v"(RB[6]), "+v"(RB[7]), \
                       "+v"(RA[0]), "+v"(RA[1]), "+v"(RA[2]), "+v"(RA[3]), "+v"(RA[4]), "+v"(RA[5]), "+v"(RA[6]), "+v"(RA[7]), \
                       "+v"(acc[0][0]), "+v"(acc[0][1]), "+v"(acc[1][0]), "+v"(acc[1][1]), \
                       "+v"(acc[2][0]), "+v"(acc[2][1]), "+v"(acc[3][0]), "+v"(acc[3][1]))
#define TAKE() do { const int rn_ = __builtin_amdgcn_readfirstlane(rv); rk = WANT() ? rn_ : rk; iss = max(iss, min(rk, r + NS)); aR ^= 4u; } while (0)
#define STEP(RB, RA, MB, MA) do { \
        const v8i w0_ = MKB(MB[0], MB[1], MB[2], MB[3]), w1_ = MKB(MB[4], MB[5], MB[6], MB[7]); \
        const v8i x0_ = MK8(MA[0], MA[1]), x1_ = MK8(MA[2], MA[3]), x2_ = MK8(MA[4], MA[5]); \
        TR8(RB[0], aB, 0); asm volatile("ds_read_b32 %0, %1" : "=v"(rv) : "v"(aR)); \
        TR8(RB[1], aB, 8320); TR8(RB[2], aB, 512); TR8(RB[3], aB, 8832); SB(); \
        MMA(acc[0][0], w0_, x0_); SB(); TR8(RB[4], aB, 16); TR8(RB[5], aB, 8336); TR8(RB[6], aB, 528); SB(); \
        MMA(acc[0][1], w1_, x0_); SB(); TR8(RB[7], aB, 8848); RD128(RA[0], aA, 0); RD128(RA[1], aA, 64); SB(); \
        MMA(acc[1][0], w0_, x1_); SB(); RD128(RA[2], aA, 256); RD128(RA[3], aA, 320); SB(); \
        MMA(acc[1][1], w1_, x1_); SB(); RD128(RA[4], aA, 512); RD128(RA[5], aA, 576); SB(); \
        MMA(acc[2][0], w0_, x2_); SB(); RD128(RA[6], aA, 768); RD128(RA[7], aA, 832); SB(); \
        MMA(acc[2][1], w1_, x2_); SB(); \
        asm volatile("s_waitcnt lgkmcnt(15)" : "+v"(rv)); \
        TAKE(); ++r; rd = r < iss; \
        { const int dlt = (slot == NS - 1) ? -(NS - 1) * SLOT : SLOT; slot = (slot == NS - 1) ? 0 : slot + 1; aA += dlt; aB += dlt; } \
        asm volatile("" : "+v"(aA), "+v"(aB), "+v"(aR)); SB(); } while (0)
#define STEPB(RB, RA, MB, MA) do { \
        const v8i w0_ = MKB(MB[0], MB[1], MB[2], MB[3]), w1_ = MKB(MB[4], MB[5], MB[6], MB[7]); \
        const v8i x3_ = MK8(MA[6], MA[7]); \
        MMA(acc[3][0], w0_, x3_); SB(); \
        MMA(acc[3][1], w1_, x3_); SB(); \
        WAITALL(RB, RA); } while (0)
    const float sc = p.sx[0] * p.sw[0];
    int r = 0, iss = 4, rk = 4;
    int slot = 0;
    unsigned aA = rdA, aB = rdB, aR = C_READY;
    bool rd = true;
    int rv;
    if (w == 0) STAMP(0);
    BARRIER();
#define PHASE(RB, RA, MB, MA) do { PHASEA(RB, RA, MB, MA); STEPB(RB, RA, MB, MA); } while (0)
#define PHASEA(RB, RA, MB, MA) do { \
      BARRIER(); \
      while (__builtin_expect(!rd, 0)) { \
        rv = lds_read_now(aR); \
        TAKE(); rd = r < iss; \
        BARRIER(); } \
      if (w == 0) STAMP(1 + r); \
      STEP(RB, RA, MB, MA); } while (0)
    for (int rr = 0; rr < NSTAGE / 2; ++rr) {
      PHASE(pb, pa, qb, qa);
      PHASE(qb, qa, pb, pa);
    }
    MMA_ALL(qb, qa);
    if (w == 0) STAMP(33);
    const unsigned bad = __builtin_amdgcn_readfirstlane(lds_read_now(C_BAD));
    float* ob = p.out + (size_t)(mb * BM + i16) * NTOT + ntile * BN + w * 32 + g * 4;
#pragma unroll
    for (int mt = 0; mt < 4; ++mt)
#pragma unroll
      for (int nt = 0; nt < 2; ++nt) {
        v4f v = acc[mt][nt];
        v4f y;
#pragma unroll
        for (int q = 0; q < 4; ++q) y[q] = bad ? __builtin_nanf("") : fmaxf(v[q] * sc, 0.f);
        __builtin_nontemporal_store(y, (v4f*)(ob + (size_t)mt * 16 * NTOT + nt * 16));
      }
  } else if (wave < 8) {
    const int w = wave - 4;
    const unsigned offA = (2 * w + 8 * (lane >> 3)) * KC + (lane & 7) * 16;
    const unsigned offS = 2 * w * 1024 + lane * 16;
    const unsigned offB = (4 * w + 16 * (lane >> 3)) * NTOT + (lane & 7) * 16 + ntile * BN;
    const unsigned char* own_a = p.x + (size_t)(rank * 512 + mb * BM) * KC;
    BARRIER();
#pragma unroll
    for (int j = 0; j < 4; ++j) {
      lds_u8* la = smem + j * SLOT + (2 * w) * IPA;
      lds_u8* lb = smem + j * SLOT + SLOT_A + (4 * w) * IPAD;
      const unsigned char* bb = p.w + (size_t)(rank * KC + j * BK) * NTOT;
#pragma unroll
      for (int i = 0; i < 2; ++i)
        __builtin_amdgcn_global_load_lds((const unsigned*)(own_a + j * BK + offA + i * KC), (lds_u32*)(la + i * IPA), 16, 0, 0);
#pragma unroll
      for (int i = 0; i < 4; ++i)
        __builtin_amdgcn_global_load_lds((const unsigned*)(bb + offB + i * NTOT), (lds_u32*)(lb + i * IPAD), 16, 0, 0);
    }
    const unsigned aQ = C_Q + min(lane, 34) * 4;
    const unsigned oS0 = offS, oS1 = offS + 1024;
    const unsigned oB0 = offB, oB1 = offB + NTOT, oB2 = offB + 2 * NTOT, oB3 = offB + 3 * NTOT;
    const unsigned lA = 2 * w * IPA, lB = SLOT_A + 4 * w * IPAD;
    int nslot = 4, qv = 0;
    const unsigned char* stg_a = nullptr;
#define GLDS(off, base, l, pol) asm volatile("s_mov_b32 m0, %2\n\ts_nop 0\n\tglobal_load_lds_dwordx4 %0, %1" pol ::"v"(off), "s"(base), "s"(l) : "memory")
#define ISSUE(ab, bb, sl) do { \
      GLDS(oS0, ab, sl + lA, " sc0 sc1"); GLDS(oS1, ab, sl + lA + IPA, " sc0 sc1"); \
      GLDS(oB0, bb, sl + lB, ""); GLDS(oB1, bb, sl + lB + IPAD, ""); \
      GLDS(oB2, bb, sl + lB + 2 * IPAD, ""); GLDS(oB3, bb, sl + lB + 3 * IPAD, ""); } while (0)
    int r = 0, iss = 4, it = 0, rk = 4;
    for (;;) {
      if (iss == r + NS - 1) {
        const int rend = rk == NSTAGE ? NSTAGE - (NS - 1) : rk - (NS + AHEAD) + 1;
        while (r < rend) {
          const int c = __builtin_amdgcn_readlane(qv, r + NS - 1);
          const unsigned char* ab = stg_a + (unsigned)c * PIECE;
          const unsigned char* bb = p.w + (unsigned)c * (BK * NTOT);
          const unsigned sl = nslot * SLOT;
          asm volatile("" ::"s"(ab), "s"(bb), "s"(sl));
          __builtin_amdgcn_sched_barrier(0);
          asm volatile("s_waitcnt vmcnt(24)\n\ts_barrier" ::: "memory");
          __builtin_amdgcn_sched_barrier(0);
          ISSUE(ab, bb, sl);
          nslot = (nslot == NS - 1) ? 0 : nslot + 1;
          ++r;
          ++it;
        }
        iss = r + NS - 1;
      }
      const int rd = r < iss ? 1 : 0;
      const int lim = min(rk, r + NS);
      const int c0 = __builtin_amdgcn_readlane(qv, iss & 31);
      const unsigned char* ab0 = stg_a + (unsigned)c0 * PIECE;
      const unsigned char* bb0 = p.w + (unsigned)c0 * (BK * NTOT);
      const unsigned sl0 = nslot * SLOT;
      asm volatile("" ::"s"(ab0), "s"(bb0), "s"(sl0));
      wait_vm(iss - r - 1);
      __builtin_amdgcn_sched_barrier(0);
      BARRIER();
      __builtin_amdgcn_sched_barrier(0);
      int qn;
      asm volatile("ds_read_b32 %0, %1" : "=v"(qn) : "v"(aQ) : "memory");
      if (iss < lim) {
        ISSUE(ab0, bb0, sl0);
        ++iss;
        nslot = (nslot == NS - 1) ? 0 : nslot + 1;
        while (iss < lim) {
          const int c = __builtin_amdgcn_readlane(qv, iss);
          const unsigned char* ab = stg_a + (unsigned)c * PIECE;
          const unsigned char* bb = p.w + (unsigned)c * (BK * NTOT);
          const unsigned sl = nslot * SLOT;
          ISSUE(ab, bb, sl);
          ++iss;
          nslot = (nslot == NS - 1) ? 0 : nslot + 1;
        }
      }
      asm volatile("s_waitcnt lgkmcnt(0)" : "+v"(qn)::"memory");
      if (WANT()) {
        qv = qn;
        rk = __builtin_amdgcn_readlane(qv, 32 + (it & 1));
        if (it == 0) {
          unsigned call = __builtin_amdgcn_readlane(qv, 34);
          stg_a = myws + WS_STAGE + (size_t)(call & 1) * STAGE_PAR + (size_t)mb * STAGE_MB;
        }
        const int lim2 = min(rk, r + NS);
        while (iss < lim2) {
          const int c = __builtin_amdgcn_readlane(qv, iss);
          const unsigned char* ab = stg_a + (unsigned)c * PIECE;
          const unsigned char* bb = p.w + (unsigned)c * (BK * NTOT);
          const unsigned sl = nslot * SLOT;
          ISSUE(ab, bb, sl);
          ++iss;
          nslot = (nslot == NS - 1) ? 0 : nslot + 1;
        }
      }
      r += rd;
      ++it;
      if (r == NSTAGE) break;
    }
  } else if (wave == 8) {
    __builtin_amdgcn_s_setprio(3);
    const int dst = b & 7, pj = b >> 3, pm = (pj + rank) & 7, ps = pj >> 3;
    const bool active = dst != rank;
    unsigned call = __hip_atomic_load(cnt_ptr, __ATOMIC_RELAXED, __HIP_MEMORY_SCOPE_AGENT) + 1u;
    unsigned* flag = (unsigned*)(p.ws[dst] + WS_FLAGS + (size_t)pm * 128 + (rank * 4 + ps) * 4);
    v4i d[8];
    {
      const unsigned char* src = p.x + (size_t)(dst * 512 + pm * BM + 8 * (lane >> 3)) * KC + ps * BK + (lane & 7) * 16;
#pragma unroll
      for (int j = 0; j < 8; ++j) d[j] = *(const v4i*)(src + j * KC);
      asm volatile("" : "+v"(d[0]), "+v"(d[1]), "+v"(d[2]), "+v"(d[3]), "+v"(d[4]), "+v"(d[5]), "+v"(d[6]), "+v"(d[7]), "+v"(call));
      asm volatile("s_waitcnt vmcnt(0)" ::: "memory");
    }
    const unsigned long long tst = __builtin_amdgcn_s_memrealtime();
    call = __builtin_amdgcn_readfirstlane(call);
    unsigned char* dp = p.ws[dst] + WS_STAGE + (size_t)(call & 1) * STAGE_PAR + (size_t)pm * STAGE_MB + (size_t)(rank * 4 + ps) * PIECE + lane * 16;
#define PUSH() do { _Pragma("unroll") for (int j = 0; j < 8; ++j) \
      asm volatile("global_store_dwordx4 %0, %1, off sc0 sc1\n\ts_nop 1" ::"v"(dp + j * 1024), "v"(d[j]) : "memory"); } while (0)
    int st = active ? 0 : 2;
    if (st == 0 && pj < EARLY) {
      STAMP(35);
      asm volatile("s_waitcnt vmcnt(0)" ::: "memory");
      PUSH();
      st = 1;
    }
    BARRIER();
    if (st == 1) {
      asm volatile("s_waitcnt vmcnt(0)" ::: "memory");
      if (lane == 0) __hip_atomic_store(flag, call, __ATOMIC_RELAXED, __HIP_MEMORY_SCOPE_SYSTEM);
      st = 2;
      STAMP(36);
    }
#define CLK_BARRIER(t) asm volatile("s_memrealtime %0\n\ts_barrier\n\ts_waitcnt lgkmcnt(0)" : "=s"(t)::"memory")
    if (st == 0) {
      const unsigned ts = (unsigned)tst + (pj < SLOT_N1 ? pj * SLOT_T1 : SLOT_N1 * SLOT_T1 + (pj - SLOT_N1) * SLOT_T2);
      unsigned long long tn = tst;
      while ((int)((unsigned)tn - ts) < -XW) CLK_BARRIER(tn);
      while ((int)((unsigned)__builtin_amdgcn_s_memrealtime() - ts) < -XS) BARRIER();
      while ((int)((unsigned)__builtin_amdgcn_s_memrealtime() - ts) < 0) __builtin_amdgcn_s_sleep(1);
      STAMP(35);
      asm volatile("s_waitcnt vmcnt(0)" ::: "memory");
      PUSH();
      do {
        CLK_BARRIER(tn);
        if ((int)((unsigned)tn - ts) >= XA - 30) asm volatile("s_waitcnt vmcnt(0)" ::: "memory");
        else for (int k = 0; k < SPINB && vm_busy(); ++k) __builtin_amdgcn_s_sleep(1);
      } while (vm_busy());
      if (lane == 0) __hip_atomic_store(flag, call, __ATOMIC_RELAXED, __HIP_MEMORY_SCOPE_SYSTEM);
      STAMP(36);
    }
    while (vm_busy()) BARRIER();
  } else {
    __builtin_amdgcn_s_setprio(3);
    unsigned call = __hip_atomic_load(cnt_ptr, __ATOMIC_RELAXED, __HIP_MEMORY_SCOPE_AGENT) + 1u;
    __builtin_amdgcn_sched_barrier(0);
    BARRIER();
    __builtin_amdgcn_sched_barrier(0);
    call = __builtin_amdgcn_readfirstlane(call);
    if (lane == 0) {
      *(volatile lds_u32*)(smem + C_CALL) = call;
      *(volatile lds_u32*)(smem + C_BAD) = 0u;
    }
    const unsigned char* fp = myws + WS_FLAGS + (size_t)mb * 128 + (lane & 31) * 4;
    const bool mine = lane < 32 && (lane >> 2) != rank;
    const unsigned long long t0 = __builtin_amdgcn_s_memrealtime();
    unsigned done = 0;
    const unsigned below = (1u << (lane & 31)) - 1u;
    int rdy = 4, npoll = 0;
    unsigned aP = C_READY;
#define PUBLISH() asm volatile("s_mov_b64 exec, 1\n\tds_write_b32 %0, %1\n\ts_mov_b64 exec, -1\n\ts_waitcnt lgkmcnt(0)" ::"v"(aP), "v"(rdy) : "memory")
#define NEXT() do { BARRIER(); aP ^= 4u; } while (0)
    asm volatile("ds_write_b32 %0, %1" ::"v"(aP ^ 4u), "v"(rdy) : "memory");
    PUBLISH();
    unsigned nw = 0;
    do {
      int v;
      if (rdy >= 16) {
        if (nw) { PUBLISH(); NEXT(); PUBLISH(); nw = 0; }
        NEXT();
      }
      asm volatile("global_load_dword %0, %1, off sc0 sc1" : "=v"(v) : "v"(fp) : "memory");
      if (nw) { PUBLISH(); NEXT(); PUBLISH(); } else NEXT();
      while (vm_busy()) NEXT();
      asm volatile("s_waitcnt vmcnt(0)" : "+v"(v)::"memory");
      bool tmo = false;
      if (__builtin_expect((++npoll & 15) == 0, 0)) tmo = __builtin_amdgcn_s_memrealtime() - t0 > DEADLINE_TICKS;
      const bool ok = mine && ((int)((unsigned)v - call) >= 0 || tmo);
      nw = (unsigned)__ballot(ok) & ~done;
      if (nw) {
        if ((nw >> (lane & 31)) & 1)
          asm volatile("ds_write_b32 %0, %1" ::"v"(C_Q + (rdy + __builtin_popcount(nw & below)) * 4), "v"(lane & 31) : "memory");
        rdy += __builtin_popcount(nw);
        done |= nw;
        if (__builtin_expect(tmo, 0) && lane == 0) {
          asm volatile("ds_write_b32 %0, %1" ::"v"(C_BAD), "v"(1) : "memory");
          __hip_atomic_store((unsigned*)(myws + WS_TMO), call, __ATOMIC_RELAXED, __HIP_MEMORY_SCOPE_AGENT);
        }
      }
    } while (rdy < NSTAGE);
    if (lane == 0) __hip_atomic_store(cnt_ptr, call, __ATOMIC_RELAXED, __HIP_MEMORY_SCOPE_AGENT);
    PUBLISH();
    NEXT();
    PUBLISH();
    while (vm_busy()) BARRIER();
  }
}
}

extern "C" void kernel_launch(int rank, int nranks, ncclComm_t comm, hipStream_t stream,
                              void* const* d_in, const int* in_sizes, int n_in,
                              void* d_out, int out_size, const int* in_dtypes, int out_dtype,
                              void* const* d_out_all, void* const* d_ws_all, size_t ws_bytes) {
  if (nranks != NR || ws_bytes < WS_TOTAL) {
    fprintf(stderr, "unsupported configuration\n");
    return;
  }
  static bool attr_set = false;
  if (!attr_set) {
    (void)hipFuncSetAttribute((const void*)fused_a2a_gemm, hipFuncAttributeMaxDynamicSharedMemorySize, LDS_BYTES);
    attr_set = true;
  }
  Params p;
  p.x = (const unsigned char*)d_in[0];
  p.w = (const unsigned char*)d_in[1];
  p.sx = (const float*)d_in[2];
  p.sw = (const float*)d_in[3];
  p.out = (float*)d_out;
  for (int i = 0; i < NR; ++i) p.ws[i] = (unsigned char*)d_ws_all[i];
  p.rank = rank;
  hipLaunchKernelGGL(fused_a2a_gemm, dim3(NWG), dim3(NTHREADS), LDS_BYTES, stream, p);
}
```
